```python
import numpy as np

kernel_name = "gpu_ag_gemm_m4096_k4096_n16384_bf16_none"


ABSMAX_RTOL = 0.008

COMPUTE_DTYPE = "bfloat16"
EPILOGUE = "none"
IN_QUANT = None
OUT_QUANT = None
IN_DTYPES = {"x": "bfloat16", "w_mat": "bfloat16"}
WORLD_SIZE = 8

def _to_bf16(x):
    u = np.ascontiguousarray(x, dtype=np.float32).view(np.uint32)
    return ((u + 0x7FFF + ((u >> 16) & 1)) & 0xFFFF0000).view(np.float32)

def setup_inputs():

    x = _to_bf16(np.random.randn(4096, 4096).astype(np.float32))
    w_mat = _to_bf16((np.random.randn(4096, 16384) / np.sqrt(4096)).astype(np.float32))
    return {"x": x, "w_mat": w_mat}

def _epilogue(y):
    return y

def reference(x, w_mat):
    y = (x @ w_mat).astype(np.float32)
    return _epilogue(y)
```

```cpp
#include <hip/hip_runtime.h>
#include <rccl/rccl.h>
#include <cstdio>

#define LAS __attribute__((address_space(3)))
typedef unsigned u32;
typedef unsigned long long u64;
typedef u32 u32x4 __attribute__((ext_vector_type(4)));
typedef u32 u32x2 __attribute__((ext_vector_type(2)));
typedef short s16x8 __attribute__((ext_vector_type(8)));
typedef short s16x4 __attribute__((ext_vector_type(4)));
typedef float f32x4 __attribute__((ext_vector_type(4)));
typedef __bf16 bf16x2 __attribute__((ext_vector_type(2)));
typedef float f32x2 __attribute__((ext_vector_type(2)));

namespace {
constexpr int NR = 8, KD = 4096, NL = 2048, ML = 512;
constexpr int NT = 64, NH = 128;
constexpr int NA = 3, NW = 6, SLOT = 16384;
constexpr int A_OFF = 0, W_OFF = NA * SLOT, C_OFF = (NA + NW) * SLOT, LDS_BYTES = C_OFF + 256;
constexpr int C_FULL = 0;
constexpr int C_FREE = 16;
constexpr int C_AVAIL = 32;
constexpr int C_BAD = 36;
constexpr size_t WS_CNT = 0;
constexpr size_t WS_TMO = 4096;
constexpr size_t WS_FLAG = 8192;
constexpr size_t WS_DATA = 1 << 20;
constexpr size_t PAR_BYTES = (size_t)32 * NT * SLOT;
constexpr size_t WS_NEED = WS_DATA + 2 * PAR_BYTES;
constexpr u64 DEADLINE = 50000000ull;

struct Params {
  const char* x; const char* w; char* out; char* ws[NR]; int rank; int pad;
};

__device__ __forceinline__ u64 rt() { return __builtin_amdgcn_s_memrealtime(); }
__device__ __forceinline__ u32 rfl(u32 v) { return __builtin_amdgcn_readfirstlane(v); }
__device__ __forceinline__ u32x4 mk_rsrc(const void* p, u32 bytes) {
  u64 a = (u64)p; u32x4 r; r.x = rfl((u32)a); r.y = rfl((u32)(a >> 32)) & 0xffffu; r.z = bytes; r.w = 0x00020000u; return r;
}
template <int SYS> __device__ __forceinline__ void glds16(u32 lds, u32 voff, u32x4 rsrc, u32 soff) {
  if (SYS) asm volatile("s_mov_b32 m0, %0\n\ts_nop 0\n\tbuffer_load_dwordx4 %1, %2, %3 offen sc0 sc1 lds" :: "s"(lds), "v"(voff), "s"(rsrc), "s"(soff) : "memory");
  else     asm volatile("s_mov_b32 m0, %0\n\ts_nop 0\n\tbuffer_load_dwordx4 %1, %2, %3 offen lds" :: "s"(lds), "v"(voff), "s"(rsrc), "s"(soff) : "memory");
}
__device__ __forceinline__ u32x4 lds_rd4(u32 addr) { u32x4 r; asm volatile("ds_read_b128 %0, %1\n\ts_waitcnt lgkmcnt(0)" : "=&v"(r) : "v"(addr) : "memory"); return r; }
__device__ __forceinline__ u32 lds_rd1(u32 addr) { u32 r; asm volatile("ds_read_b32 %0, %1\n\ts_waitcnt lgkmcnt(0)" : "=&v"(r) : "v"(addr) : "memory"); return r; }
__device__ __forceinline__ void lds_wr1(u32 addr, u32 v) { asm volatile("ds_write_b32 %0, %1" :: "v"(addr), "v"(v) : "memory"); }
__device__ __forceinline__ u32 min_free(u32 cbase) { u32x4 f = lds_rd4(cbase + C_FREE); return rfl(min(min(f.x, f.y), min(f.z, f.w))); }
#define WAIT_VM(n) asm volatile("s_waitcnt vmcnt(" #n ")" ::: "memory")
__device__ __forceinline__ void st16_sys(char* p, u32x4 v) { asm volatile("global_store_dwordx4 %0, %1, off sc0 sc1\n\ts_nop 1" :: "v"(p), "v"(v) : "memory"); }
__device__ __forceinline__ u32 pk_bf16(float a, float b) { f32x2 f = {a, b}; bf16x2 h = __builtin_convertvector(f, bf16x2); return __builtin_bit_cast(u32, h); }
}

__global__ void __launch_bounds__(512, 2) ag_gemm_kernel(Params P) {
  extern __shared__ __attribute__((aligned(16))) char smem[];
  const u32 lbase = (u32)(size_t)(LAS char*)smem;
  const u32 cbase = lbase + C_OFF;
  const int tid = threadIdx.x, lane = tid & 63;
  const int wave = rfl(tid >> 6);
  const int cb = blockIdx.x & 7, rb = blockIdx.x >> 3;
  const int src = rb >> 2;
  const bool own = (src == P.rank);
  char* myws = P.ws[P.rank];
  u32* cnt = (u32*)(myws + WS_CNT) + blockIdx.x;
  const u32 call = rfl(__hip_atomic_load(cnt, __ATOMIC_RELAXED, __HIP_MEMORY_SCOPE_AGENT)) + 1u;
  if (tid < 64) ((LAS u32*)(smem + C_OFF))[tid] = 0u;
  __syncthreads();
  if (tid == 0) __hip_atomic_store(cnt, call, __ATOMIC_RELAXED, __HIP_MEMORY_SCOPE_AGENT);
  const size_t par_off = WS_DATA + (size_t)(call & 1u) * PAR_BYTES;

  if (wave < 4) {
    __builtin_amdgcn_s_setprio(2);
    const int w = wave;
    const u32 r16 = lane & 15, g = lane >> 4, q = (lane >> 2) & 3, p = lane & 3;
    const u32 xoff = lbase + A_OFF + r16 * 128 + 16 * (g ^ (r16 >> 1));
    const u32 sw = (q >> 1) | ((g & 1) << 1);
    u32 woff[4];
#pragma unroll
    for (int nf = 0; nf < 4; ++nf) woff[nf] = lbase + W_OFF + w * 4096 + g * 1024 + q * 128 + ((nf ^ sw) * 32) + 8 * p;
    f32x4 acc[8][4];
#pragma unroll
    for (int m = 0; m < 8; ++m)
#pragma unroll
      for (int n = 0; n < 4; ++n) acc[m][n] = (f32x4){0.f, 0.f, 0.f, 0.f};
    s16x8 X[8], W0[4], W1[4];
    auto rdX = [&](u32 base, int m) -> s16x8 { return *(const LAS s16x8*)(size_t)(base + m * 2048); };
    auto rdW = [&](u32 base) -> s16x8 {
      s16x4 lo = __builtin_amdgcn_ds_read_tr16_b64_v4i16((LAS s16x4*)(size_t)(base));
      s16x4 hi = __builtin_amdgcn_ds_read_tr16_b64_v4i16((LAS s16x4*)(size_t)(base + 512));
      return __builtin_shufflevector(lo, hi, 0, 1, 2, 3, 4, 5, 6, 7);
    };
    auto wait_full = [&](u32 s) {
      for (;;) {
        u32x2 f = *(volatile LAS u32x2*)(size_t)(cbase + C_FULL);
        if (rfl(f.x) > s && rfl(f.y) > (s >> 1)) break;
        __builtin_amdgcn_s_sleep(1);
      }
      asm volatile("" ::: "memory");
    };
    u32 aslot = 0, wslot = 0;
    wait_full(0);
#pragma unroll
    for (int n = 0; n < 4; ++n) W0[n] = rdW(woff[n]);
#pragma unroll
    for (int m = 0; m < 8; ++m) X[m] = rdX(xoff, m);
    lds_wr1(cbase + C_FREE + 4 * w, 1u);
#define X1 X
#define HALF_STEP(WC, WN, XC, XN, XBASE, LASTFLAG) do { \
      if (!(LASTFLAG)) { _Pragma("unroll") for (int n = 0; n < 4; ++n) WN[n] = rdW(woff[n] + wslot); } \
      _Pragma("unroll") for (int m = 0; m < 8; ++m) { \
        _Pragma("unroll") for (int n = 0; n < 4; ++n) acc[m][n] = __builtin_amdgcn_mfma_f32_16x16x32_bf16(WC[n], X[m], acc[m][n], 0, 0, 0); \
        if (!(LASTFLAG)) X[m] = rdX(XBASE, m); } \
      if (!(LASTFLAG)) { \
        __builtin_amdgcn_sched_group_barrier(0x100, 2, 0); \
        _Pragma("unroll") for (int m = 0; m < 3; ++m) { __builtin_amdgcn_sched_group_barrier(0x8, 4, 0); __builtin_amdgcn_sched_group_barrier(0x100, 3, 0); } \
        _Pragma("unroll") for (int m = 3; m < 8; ++m) { __builtin_amdgcn_sched_group_barrier(0x8, 4, 0); __builtin_amdgcn_sched_group_barrier(0x100, 1, 0); } } \
    } while (0)
    for (int t = 0; t < NT - 1; ++t) {
      const u32 s = 2 * t;
      wslot += SLOT; if (wslot == NW * SLOT) wslot = 0;
      wait_full(s + 1);
      HALF_STEP(W0, W1, X, X1, (xoff ^ 64u) + aslot, false);
      lds_wr1(cbase + C_FREE + 4 * w, s + 2);
      wslot += SLOT; if (wslot == NW * SLOT) wslot = 0;
      aslot += SLOT; if (aslot == NA * SLOT) aslot = 0;
      wait_full(s + 2);
      HALF_STEP(W1, W0, X1, X, xoff + aslot, false);
      lds_wr1(cbase + C_FREE + 4 * w, s + 3);
    }
    wslot += SLOT; if (wslot == NW * SLOT) wslot = 0;
    wait_full(NH - 1);
    HALF_STEP(W0, W1, X, X1, (xoff ^ 64u) + aslot, false);
    HALF_STEP(W1, W0, X1, X, 0u, true);
    const bool bad = lds_rd1(cbase + C_BAD) != 0u;
    char* orow = P.out + ((size_t)(rb * 128 + r16) * NL + cb * 256 + w * 64 + (g & 1) * 16 + (g >> 1) * 8) * 2;
#pragma unroll
    for (int m = 0; m < 8; ++m)
#pragma unroll
      for (int a = 0; a < 2; ++a) {
        u32 d0 = pk_bf16(acc[m][2 * a][0], acc[m][2 * a][1]), d1 = pk_bf16(acc[m][2 * a][2], acc[m][2 * a][3]);
        u32 e0 = pk_bf16(acc[m][2 * a + 1][0], acc[m][2 * a + 1][1]), e1 = pk_bf16(acc[m][2 * a + 1][2], acc[m][2 * a + 1][3]);
        auto r0 = __builtin_amdgcn_permlane16_swap(d0, e0, false, false);
        auto r1 = __builtin_amdgcn_permlane16_swap(d1, e1, false, false);
        u32x4 o = {r0[0], r1[0], r0[1], r1[1]};
        if (bad) o = (u32x4){0x7fc07fc0u, 0x7fc07fc0u, 0x7fc07fc0u, 0x7fc07fc0u};
        *(u32x4*)(orow + (size_t)m * 16 * NL * 2 + a * 64) = o;
      }
  } else if (wave == 4) {
    const u32x4 rsrc = mk_rsrc(P.w, (u32)KD * NL * 2);
    const u32 r = lane >> 3;
    const u32 v0 = r * (NL * 2) + ((((lane & 7) >> 1) ^ ((r & 3) >> 1)) * 32) + (lane & 1) * 16, v1 = v0 ^ 64u;
    u32 slot = 0, known = 0, pub = 0;
    for (u32 s = 0; s < NH; ++s) {
      if (s >= NW && known < s - NW + 1) {
        known = min_free(cbase);
        if (known < s - NW + 1) {
          WAIT_VM(0); lds_wr1(cbase + C_FULL, s); pub = s;
          do { __builtin_amdgcn_s_sleep(1); known = min_free(cbase); } while (known < s - NW + 1);
        }
      }
      const u32 soff = s * 32 * (NL * 2) + cb * 512, ld = lbase + W_OFF + slot;
#pragma unroll
      for (int gg = 0; gg < 4; ++gg)
#pragma unroll
        for (int ww = 0; ww < 4; ++ww) glds16<0>(ld + ww * 4096 + gg * 1024, (gg & 1) ? v1 : v0, rsrc, soff + gg * 8 * (NL * 2) + ww * 128);
      slot += SLOT; if (slot == NW * SLOT) slot = 0;
      if (pub + 1 < s) { WAIT_VM(32); pub = s - 1; lds_wr1(cbase + C_FULL, pub); }
    }
    WAIT_VM(0); lds_wr1(cbase + C_FULL, NH);
  } else if (wave == 5) {
    const u32 r = lane >> 3;
    u32x4 rsrc; u32 v0, v1, istep, tstep, sbase;
    if (own) {
      rsrc = mk_rsrc(P.x, (u32)ML * KD * 2);
      v0 = r * (KD * 2) + 16 * ((lane & 7) ^ (lane >> 4)); v1 = v0 ^ 64u; istep = 8 * KD * 2; tstep = 128; sbase = (rb & 3) * 128 * KD * 2;
    } else {
      rsrc = mk_rsrc(myws + par_off + (size_t)rb * NT * SLOT, (u32)NT * SLOT);
      v0 = lane * 16; v1 = v0; istep = 1024; tstep = SLOT; sbase = 0;
    }
    u32 slot = 0, known = 0, avail = own ? NT : 0, pub = 0;
    for (u32 t = 0; t < NT; ++t) {
      const u32 need = t >= NA ? 2 * (t - NA) + 2 : 0;
      if (known < need) known = min_free(cbase);
      if (avail <= t) avail = rfl(lds_rd1(cbase + C_AVAIL));
      if (known < need || avail <= t) {
        WAIT_VM(0); lds_wr1(cbase + C_FULL + 4, t); pub = t;
        while (avail <= t) { __builtin_amdgcn_s_sleep(1); avail = rfl(lds_rd1(cbase + C_AVAIL)); }
        while (known < need) { __builtin_amdgcn_s_sleep(1); known = min_free(cbase); }
      }
      const u32 soff = sbase + t * tstep, ld = lbase + A_OFF + slot;
#pragma unroll
      for (int i = 0; i < 16; ++i) glds16<1>(ld + i * 1024, (i & 1) ? v1 : v0, rsrc, soff + i * istep);
      slot += SLOT; if (slot == NA * SLOT) slot = 0;
      if (pub < t) { WAIT_VM(16); lds_wr1(cbase + C_FULL + 4, t); pub = t; }
    }
    WAIT_VM(0); lds_wr1(cbase + C_FULL + 4, NT);
  } else if (wave == 6 && !own) {
    const u32* fl = (const u32*)(myws + WS_FLAG + (size_t)rb * 128) + (lane & 3);
    u32* tmo = (u32*)(myws + WS_TMO);
    const u32 fbase = call * 128u;
    u32 cur = 0; const u64 t0 = rt(); u32 it = 0;
    while (cur < NT) {
      u32 v = __hip_atomic_load(fl, __ATOMIC_RELAXED, __HIP_MEMORY_SCOPE_SYSTEM);
      u32 d = (int)(v - fbase) > 0 ? min(v - fbase, (u32)NT) : 0u;
      u32 m = min(min(__builtin_amdgcn_readlane(d, 0), __builtin_amdgcn_readlane(d, 1)), min(__builtin_amdgcn_readlane(d, 2), __builtin_amdgcn_readlane(d, 3)));
      if (m > cur) { cur = m; lds_wr1(cbase + C_AVAIL, cur); continue; }
      if ((++it & 63u) == 0u) {
        if (rfl(__hip_atomic_load(tmo, __ATOMIC_RELAXED, __HIP_MEMORY_SCOPE_AGENT)) == call || rt() - t0 > DEADLINE) {
          __hip_atomic_store(tmo, call, __ATOMIC_RELAXED, __HIP_MEMORY_SCOPE_AGENT);
          lds_wr1(cbase + C_BAD, 1u); lds_wr1(cbase + C_AVAIL, NT); break;
        }
      }
      __builtin_amdgcn_s_sleep(2);
    }
  } else if (wave == 7 && own && (rb & 1) == 0) {
    const int rg = ((rb & 3) >> 1) * 8 + cb;
    const char* sp = P.x + (size_t)(rg * 32 + (lane >> 3)) * (KD * 2);
    const u32 c0 = 16 * ((lane & 7) ^ (lane >> 4)), c1 = c0 ^ 64u;
    const char* s0 = sp + c0; const char* s1 = sp + 8 * KD * 2 + c1; const char* s2 = sp + 16 * KD * 2 + c0; const char* s3 = sp + 24 * KD * 2 + c1;
    const size_t doff = par_off + (size_t)(P.rank * 4 + (rg >> 2)) * NT * SLOT + (rg & 3) * 4096 + lane * 16;
    char* dst[7];
#pragma unroll
    for (int j = 0; j < 7; ++j) { u32 pp = ((u32)P.rank + 1u + (u32)(j + rg) % 7u) & 7u; dst[j] = P.ws[pp] + doff; }
    const u32 pl = ((u32)P.rank + 1u + (u32)lane % 7u) & 7u;
    char* fl = P.ws[pl] + WS_FLAG + (size_t)(P.rank * 4 + (rg >> 2)) * 128 + (rg & 3) * 4;
    const u32 fbase = call * 128u;
    u32x4 v[4];
#define LD4(T, EXTRA) asm volatile("global_load_dwordx4 %0, %4, off\n\tglobal_load_dwordx4 %1, %5, off\n\tglobal_load_dwordx4 %2, %6, off\n\tglobal_load_dwordx4 %3, %7, off" EXTRA \
      : "=&v"(v[0]), "=&v"(v[1]), "=&v"(v[2]), "=&v"(v[3]) : "v"(s0 + (T) * 128), "v"(s1 + (T) * 128), "v"(s2 + (T) * 128), "v"(s3 + (T) * 128) : "memory")
    LD4(0, "\n\ts_waitcnt vmcnt(0)");
    for (int t = 0; t < NT; ++t) {
      u32x4 c[4];
#pragma unroll
      for (int i = 0; i < 4; ++i) c[i] = v[i];
      asm volatile("" : "+v"(c[0]), "+v"(c[1]), "+v"(c[2]), "+v"(c[3]));
      const int tn = t + 1 < NT ? t + 1 : t;
      LD4(tn, "");
#pragma unroll
      for (int j = 0; j < 7; ++j)
#pragma unroll
        for (int i = 0; i < 4; ++i) st16_sys(dst[j] + (size_t)t * SLOT + i * 1024, c[i]);
      asm volatile("s_waitcnt vmcnt(28)" : "+v"(v[0]), "+v"(v[1]), "+v"(v[2]), "+v"(v[3]) :: "memory");
      if (lane < 7 && t > 0) asm volatile("global_store_dword %0, %1, off sc0 sc1" :: "v"(fl), "v"(fbase + t) : "memory");
    }
    WAIT_VM(0);
    if (lane < 7) asm volatile("global_store_dword %0, %1, off sc0 sc1" :: "v"(fl), "v"(fbase + NT) : "memory");
  }
}

extern "C" void kernel_launch(int rank, int nranks, ncclComm_t comm, hipStream_t stream,
                              void* const* d_in, const int* in_sizes, int n_in, void* d_out, int out_size,
                              const int* in_dtypes, int out_dtype, void* const* d_out_all,
                              void* const* d_ws_all, size_t ws_bytes) {
  if (nranks != NR || ws_bytes < WS_NEED || (unsigned)rank >= (unsigned)NR) { fprintf(stderr, "unexpected configuration\n"); return; }
  hipError_t e = hipFuncSetAttribute((const void*)ag_gemm_kernel, hipFuncAttributeMaxDynamicSharedMemorySize, LDS_BYTES);
  if (e != hipSuccess) { fprintf(stderr, "attr: %s\n", hipGetErrorString(e)); return; }
  Params P;
  P.x = (const char*)d_in[0]; P.w = (const char*)d_in[1]; P.out = (char*)d_out; P.rank = rank; P.pad = 0;
  for (int p = 0; p < NR; ++p) P.ws[p] = (char*)d_ws_all[p];
  hipLaunchKernelGGL(ag_gemm_kernel, dim3(256), dim3(512), LDS_BYTES, stream, P);
}
```

```cpp
#include <hip/hip_runtime.h>
#include <rccl/rccl.h>
#include <cstdio>
#define CONS_ASM9 \
  "buffer_load_dword %[pd], %[pv], %[rw], %[po] offen\n\t" \
  "s_add_u32 %[po], %[po], %[pi]\n\t" \
  "buffer_load_dword %[pd], %[pv], %[rw], %[po] offen\n\t" \
  "s_add_u32 %[po], %[po], %[pi]\n\t" \
  "v_mov_b32 v112, 0\n\t" \
  "v_mov_b32 v113, 0\n\t" \
  "v_mov_b32 v114, 0\n\t" \
  "v_mov_b32 v115, 0\n\t" \
  "v_mov_b32 v116, 0\n\t" \
  "v_mov_b32 v117, 0\n\t" \
  "v_mov_b32 v118, 0\n\t" \
  "v_mov_b32 v119, 0\n\t" \
  "v_mov_b32 v120, 0\n\t" \
  "v_mov_b32 v121, 0\n\t" \
  "v_mov_b32 v122, 0\n\t" \
  "v_mov_b32 v123, 0\n\t" \
  "v_mov_b32 v124, 0\n\t" \
  "v_mov_b32 v125, 0\n\t" \
  "v_mov_b32 v126, 0\n\t" \
  "v_mov_b32 v127, 0\n\t" \
  "v_mov_b32 v128, 0\n\t" \
  "v_mov_b32 v129, 0\n\t" \
  "v_mov_b32 v130, 0\n\t" \
  "v_mov_b32 v131, 0\n\t" \
  "v_mov_b32 v132, 0\n\t" \
  "v_mov_b32 v133, 0\n\t" \
  "v_mov_b32 v134, 0\n\t" \
  "v_mov_b32 v135, 0\n\t" \
  "v_mov_b32 v136, 0\n\t" \
  "v_mov_b32 v137, 0\n\t" \
  "v_mov_b32 v138, 0\n\t" \
  "v_mov_b32 v139, 0\n\t" \
  "v_mov_b32 v140, 0\n\t" \
  "v_mov_b32 v141, 0\n\t" \
  "v_mov_b32 v142, 0\n\t" \
  "v_mov_b32 v143, 0\n\t" \
  "v_mov_b32 v144, 0\n\t" \
  "v_mov_b32 v145, 0\n\t" \
  "v_mov_b32 v146, 0\n\t" \
  "v_mov_b32 v147, 0\n\t" \
  "v_mov_b32 v148, 0\n\t" \
  "v_mov_b32 v149, 0\n\t" \
  "v_mov_b32 v150, 0\n\t" \
  "v_mov_b32 v151, 0\n\t" \
  "v_mov_b32 v152, 0\n\t" \
  "v_mov_b32 v153, 0\n\t" \
  "v_mov_b32 v154, 0\n\t" \
  "v_mov_b32 v155, 0\n\t" \
  "v_mov_b32 v156, 0\n\t" \
  "v_mov_b32 v157, 0\n\t" \
  "v_mov_b32 v158, 0\n\t" \
  "v_mov_b32 v159, 0\n\t" \
  "v_mov_b32 v160, 0\n\t" \
  "v_mov_b32 v161, 0\n\t" \
  "v_mov_b32 v162, 0\n\t" \
  "v_mov_b32 v163, 0\n\t" \
  "v_mov_b32 v164, 0\n\t" \
  "v_mov_b32 v165, 0\n\t" \
  "v_mov_b32 v166, 0\n\t" \
  "v_mov_b32 v167, 0\n\t" \
  "v_mov_b32 v168, 0\n\t" \
  "v_mov_b32 v169, 0\n\t" \
  "v_mov_b32 v170, 0\n\t" \
  "v_mov_b32 v171, 0\n\t" \
  "v_mov_b32 v172, 0\n\t" \
  "v_mov_b32 v173, 0\n\t" \
  "v_mov_b32 v174, 0\n\t" \
  "v_mov_b32 v175, 0\n\t" \
  "v_mov_b32 v176, 0\n\t" \
  "v_mov_b32 v177, 0\n\t" \
  "v_mov_b32 v178, 0\n\t" \
  "v_mov_b32 v179, 0\n\t" \
  "v_mov_b32 v180, 0\n\t" \
  "v_mov_b32 v181, 0\n\t" \
  "v_mov_b32 v182, 0\n\t" \
  "v_mov_b32 v183, 0\n\t" \
  "v_mov_b32 v184, 0\n\t" \
  "v_mov_b32 v185, 0\n\t" \
  "v_mov_b32 v186, 0\n\t" \
  "v_mov_b32 v187, 0\n\t" \
  "v_mov_b32 v188, 0\n\t" \
  "v_mov_b32 v189, 0\n\t" \
  "v_mov_b32 v190, 0\n\t" \
  "v_mov_b32 v191, 0\n\t" \
  "v_mov_b32 v192, 0\n\t" \
  "v_mov_b32 v193, 0\n\t" \
  "v_mov_b32 v194, 0\n\t" \
  "v_mov_b32 v195, 0\n\t" \
  "v_mov_b32 v196, 0\n\t" \
  "v_mov_b32 v197, 0\n\t" \
  "v_mov_b32 v198, 0\n\t" \
  "v_mov_b32 v199, 0\n\t" \
  "v_mov_b32 v200, 0\n\t" \
  "v_mov_b32 v201, 0\n\t" \
  "v_mov_b32 v202, 0\n\t" \
  "v_mov_b32 v203, 0\n\t" \
  "v_mov_b32 v204, 0\n\t" \
  "v_mov_b32 v205, 0\n\t" \
  "v_mov_b32 v206, 0\n\t" \
  "v_mov_b32 v207, 0\n\t" \
  "v_mov_b32 v208, 0\n\t" \
  "v_mov_b32 v209, 0\n\t" \
  "v_mov_b32 v210, 0\n\t" \
  "v_mov_b32 v211, 0\n\t" \
  "v_mov_b32 v212, 0\n\t" \
  "v_mov_b32 v213, 0\n\t" \
  "v_mov_b32 v214, 0\n\t" \
  "v_mov_b32 v215, 0\n\t" \
  "v_mov_b32 v216, 0\n\t" \
  "v_mov_b32 v217, 0\n\t" \
  "v_mov_b32 v218, 0\n\t" \
  "v_mov_b32 v219, 0\n\t" \
  "v_mov_b32 v220, 0\n\t" \
  "v_mov_b32 v221, 0\n\t" \
  "v_mov_b32 v222, 0\n\t" \
  "v_mov_b32 v223, 0\n\t" \
  "v_mov_b32 v224, 0\n\t" \
  "v_mov_b32 v225, 0\n\t" \
  "v_mov_b32 v226, 0\n\t" \
  "v_mov_b32 v227, 0\n\t" \
  "v_mov_b32 v228, 0\n\t" \
  "v_mov_b32 v229, 0\n\t" \
  "v_mov_b32 v230, 0\n\t" \
  "v_mov_b32 v231, 0\n\t" \
  "v_mov_b32 v232, 0\n\t" \
  "v_mov_b32 v233, 0\n\t" \
  "v_mov_b32 v234, 0\n\t" \
  "v_mov_b32 v235, 0\n\t" \
  "v_mov_b32 v236, 0\n\t" \
  "v_mov_b32 v237, 0\n\t" \
  "v_mov_b32 v238, 0\n\t" \
  "v_mov_b32 v239, 0\n\t" \
  "v_mov_b32 v240, 0\n\t" \
  "v_mov_b32 v241, 0\n\t" \
  "v_mov_b32 v242, 0\n\t" \
  "v_mov_b32 v243, 0\n\t" \
  "v_mov_b32 v244, 0\n\t" \
  "v_mov_b32 v245, 0\n\t" \
  "v_mov_b32 v246, 0\n\t" \
  "v_mov_b32 v247, 0\n\t" \
  "v_mov_b32 v248, 0\n\t" \
  "v_mov_b32 v249, 0\n\t" \
  "v_mov_b32 v250, 0\n\t" \
  "v_mov_b32 v251, 0\n\t" \
  "v_mov_b32 v252, 0\n\t" \
  "v_mov_b32 v253, 0\n\t" \
  "v_mov_b32 v254, 0\n\t" \
  "v_mov_b32 v255, 0\n\t" \
  "v_mov_b32 v28, 0\n\t" \
  "v_mov_b32 v29, 0\n\t" \
  "v_mov_b32 v30, 0\n\t" \
  "v_mov_b32 v31, 0\n\t" \
  "s_barrier\n\t" \
  "ds_read_b64_tr_b16 v[44:45], %[wl0] offset:0\n\t" \
  "ds_read_b64_tr_b16 v[46:47], %[wl0] offset:512\n\t" \
  "ds_read_b64_tr_b16 v[48:49], %[wl1] offset:0\n\t" \
  "ds_read_b64_tr_b16 v[50:51], %[wl1] offset:512\n\t" \
  "ds_read_b64_tr_b16 v[52:53], %[wl2] offset:0\n\t" \
  "ds_read_b64_tr_b16 v[54:55], %[wl2] offset:512\n\t" \
  "ds_read_b64_tr_b16 v[56:57], %[wl3] offset:0\n\t" \
  "ds_read_b64_tr_b16 v[58:59], %[wl3] offset:512\n\t" \
  "ds_read_b128 v[76:79], %[xa0] offset:0\n\t" \
  "ds_read_b128 v[80:83], %[xa0] offset:2048\n\t" \
  "ds_read_b128 v[84:87], %[xa0] offset:4096\n\t" \
  "ds_read_b128 v[88:91], %[xa0] offset:6144\n\t" \
  "ds_read_b128 v[92:95], %[xa0] offset:8192\n\t" \
  "ds_read_b128 v[96:99], %[xa0] offset:10240\n\t" \
  "ds_read_b128 v[100:103], %[xa0] offset:12288\n\t" \
  "ds_read_b128 v[104:107], %[xa0] offset:14336\n\t" \
  "ds_read_b128 v[108:111], %[xa0] offset:16384\n\t" \
  "Lloop_%=:\n\t" \
  "s_waitcnt lgkmcnt(0)\n\t" \
  "v_mfma_f32_16x16x32_bf16 v[112:115], v[44:47], v[76:79], v[112:115]\n\t" \
  "v_mfma_f32_16x16x32_bf16 v[116:119], v[48:51], v[76:79], v[116:119]\n\t" \
  "ds_read_b64_tr_b16 v[60:61], %[wl0] offset:16384\n\t" \
  "v_mfma_f32_16x16x32_bf16 v[120:123], v[52:55], v[76:79], v[120:123]\n\t" \
  "ds_read_b64_tr_b16 v[62:63], %[wl0] offset:16896\n\t" \
  "v_mfma_f32_16x16x32_bf16 v[124:127], v[56:59], v[76:79], v[124:127]\n\t" \
  "ds_read_b128 v[76:79], %[xa1] offset:0\n\t" \
  "v_mfma_f32_16x16x32_bf16 v[128:131], v[44:47], v[80:83], v[128:131]\n\t" \
  "v_mfma_f32_16x16x32_bf16 v[132:135], v[48:51], v[80:83], v[132:135]\n\t" \
  "ds_read_b64_tr_b16 v[64:65], %[wl1] offset:16384\n\t" \
  "v_mfma_f32_16x16x32_bf16 v[136:139], v[52:55], v[80:83], v[136:139]\n\t" \
  "ds_read_b64_tr_b16 v[66:67], %[wl1] offset:16896\n\t" \
  "v_mfma_f32_16x16x32_bf16 v[140:143], v[56:59], v[80:83], v[140:143]\n\t" \
  "ds_read_b128 v[80:83], %[xa1] offset:2048\n\t" \
  "v_mfma_f32_16x16x32_bf16 v[144:147], v[44:47], v[84:87], v[144:147]\n\t" \
  "v_mfma_f32_16x16x32_bf16 v[148:151], v[48:51], v[84:87], v[148:151]\n\t" \
  "ds_read_b64_tr_b16 v[68:69], %[wl2] offset:16384\n\t" \
  "v_mfma_f32_16x16x32_bf16 v[152:155], v[52:55], v[84:87], v[152:155]\n\t" \
  "ds_read_b64_tr_b16 v[70:71], %[wl2] offset:16896\n\t" \
  "v_mfma_f32_16x16x32_bf16 v[156:159], v[56:59], v[84:87], v[156:159]\n\t" \
  "ds_read_b128 v[84:87], %[xa1] offset:4096\n\t" \
  "v_mfma_f32_16x16x32_bf16 v[160:163], v[44:47], v[88:91], v[160:163]\n\t" \
  "v_mfma_f32_16x16x32_bf16 v[164:167], v[48:51], v[88:91], v[164:167]\n\t" \
  "ds_read_b64_tr_b16 v[72:73], %[wl3] offset:16384\n\t" \
  "v_mfma_f32_16x16x32_bf16 v[168:171], v[52:55], v[88:91], v[168:171]\n\t" \
  "ds_read_b64_tr_b16 v[74:75], %[wl3] offset:16896\n\t" \
  "v_mfma_f32_16x16x32_bf16 v[172:175], v[56:59], v[88:91], v[172:175]\n\t" \
  "ds_read_b128 v[88:91], %[xa1] offset:6144\n\t" \
  "v_mfma_f32_16x16x32_bf16 v[176:179], v[44:47], v[92:95], v[176:179]\n\t" \
  "v_mfma_f32_16x16x32_bf16 v[180:183], v[48:51], v[92:95], v[180:183]\n\t" \
  "ds_read_b128 v[36:39], %[xa1] offset:14336\n\t" \
  "v_mfma_f32_16x16x32_bf16 v[184:187], v[52:55], v[92:95], v[184:187]\n\t" \
  "v_mfma_f32_16x16x32_bf16 v[188:191], v[56:59], v[92:95], v[188:191]\n\t" \
  "ds_read_b128 v[92:95], %[xa1] offset:8192\n\t" \
  "v_mfma_f32_16x16x32_bf16 v[192:195], v[44:47], v[96:99], v[192:195]\n\t" \
  "v_mfma_f32_16x16x32_bf16 v[196:199], v[48:51], v[96:99], v[196:199]\n\t" \
  "ds_read_b128 v[40:43], %[xa1] offset:16384\n\t" \
  "v_mfma_f32_16x16x32_bf16 v[200:203], v[52:55], v[96:99], v[200:203]\n\t" \
  "v_mfma_f32_16x16x32_bf16 v[204:207], v[56:59], v[96:99], v[204:207]\n\t" \
  "ds_read_b128 v[96:99], %[xa1] offset:10240\n\t" \
  "v_mfma_f32_16x16x32_bf16 v[208:211], v[44:47], v[100:103], v[208:211]\n\t" \
  "v_mfma_f32_16x16x32_bf16 v[212:215], v[48:51], v[100:103], v[212:215]\n\t" \
  "v_mfma_f32_16x16x32_bf16 v[216:219], v[52:55], v[100:103], v[216:219]\n\t" \
  "v_mfma_f32_16x16x32_bf16 v[220:223], v[56:59], v[100:103], v[220:223]\n\t" \
  "ds_read_b128 v[100:103], %[xa1] offset:12288\n\t" \
  "v_mfma_f32_16x16x32_bf16 v[224:227], v[44:47], v[104:107], v[224:227]\n\t" \
  "buffer_load_dword %[pd], %[pv], %[rw], %[po] offen\n\t" \
  "s_add_u32 %[po], %[po], %[pi]\n\t" \
  "v_mfma_f32_16x16x32_bf16 v[228:231], v[48:51], v[104:107], v[228:231]\n\t" \
  "v_mfma_f32_16x16x32_bf16 v[232:235], v[52:55], v[104:107], v[232:235]\n\t" \
  "v_mfma_f32_16x16x32_bf16 v[236:239], v[56:59], v[104:107], v[236:239]\n\t" \
  "v_mfma_f32_16x16x32_bf16 v[240:243], v[44:47], v[108:111], v[240:243]\n\t" \
  "v_mfma_f32_16x16x32_bf16 v[244:247], v[48:51], v[108:111], v[244:247]\n\t" \
  "v_mfma_f32_16x16x32_bf16 v[248:251], v[52:55], v[108:111], v[248:251]\n\t" \
  "v_mfma_f32_16x16x32_bf16 v[252:255], v[56:59], v[108:111], v[252:255]\n\t" \
  "s_waitcnt lgkmcnt(0)\n\t" \
  "v_mfma_f32_16x16x32_bf16 v[112:115], v[60:63], v[76:79], v[112:115]\n\t" \
  "s_barrier\n\t" \
  "v_mfma_f32_16x16x32_bf16 v[116:119], v[64:67], v[76:79], v[116:119]\n\t" \
  "ds_read_b64_tr_b16 v[44:45], %[wl0] offset:32768\n\t" \
  "v_mfma_f32_16x16x32_bf16 v[120:123], v[68:71], v[76:79], v[120:123]\n\t" \
  "ds_read_b64_tr_b16 v[46:47], %[wl0] offset:33280\n\t" \
  "v_mfma_f32_16x16x32_bf16 v[124:127], v[72:75], v[76:79], v[124:127]\n\t" \
  "ds_read_b128 v[76:79], %[xa0] offset:20480\n\t" \
  "v_mfma_f32_16x16x32_bf16 v[128:131], v[60:63], v[80:83], v[128:131]\n\t" \
  "v_mfma_f32_16x16x32_bf16 v[132:135], v[64:67], v[80:83], v[132:135]\n\t" \
  "ds_read_b64_tr_b16 v[48:49], %[wl1] offset:32768\n\t" \
  "v_mfma_f32_16x16x32_bf16 v[136:139], v[68:71], v[80:83], v[136:139]\n\t" \
  "ds_read_b64_tr_b16 v[50:51], %[wl1] offset:33280\n\t" \
  "v_mfma_f32_16x16x32_bf16 v[140:143], v[72:75], v[80:83], v[140:143]\n\t" \
  "ds_read_b128 v[80:83], %[xa0] offset:22528\n\t" \
  "v_mfma_f32_16x16x32_bf16 v[144:147], v[60:63], v[84:87], v[144:147]\n\t" \
  "v_mfma_f32_16x16x32_bf16 v[148:151], v[64:67], v[84:87], v[148:151]\n\t" \
  "ds_read_b64_tr_b16 v[52:53], %[wl2] offset:32768\n\t" \
  "v_mfma_f32_16x16x32_bf16 v[152:155], v[68:71], v[84:87], v[152:155]\n\t" \
  "ds_read_b64_tr_b16 v[54:55], %[wl2] offset:33280\n\t" \
  "v_mfma_f32_16x16x32_bf16 v[156:159], v[72:75], v[84:87], v[156:159]\n\t" \
  "ds_read_b128 v[84:87], %[xa0] offset:24576\n\t" \
  "v_mfma_f32_16x16x32_bf16 v[160:163], v[60:63], v[88:91], v[160:163]\n\t" \
  "v_mfma_f32_16x16x32_bf16 v[164:167], v[64:67], v[88:91], v[164:167]\n\t" \
  "ds_read_b64_tr_b16 v[56:57], %[wl3] offset:32768\n\t" \
  "v_mfma_f32_16x16x32_bf16 v[168:171], v[68:71], v[88:91], v[168:171]\n\t" \
  "ds_read_b64_tr_b16 v[58:59], %[wl3] offset:33280\n\t" \
  "v_mfma_f32_16x16x32_bf16 v[172:175], v[72:75], v[88:91], v[172:175]\n\t" \
  "ds_read_b128 v[88:91], %[xa0] offset:26624\n\t" \
  "v_mfma_f32_16x16x32_bf16 v[176:179], v[60:63], v[92:95], v[176:179]\n\t" \
  "v_mfma_f32_16x16x32_bf16 v[180:183], v[64:67], v[92:95], v[180:183]\n\t" \
  "ds_read_b128 v[104:107], %[xa0] offset:34816\n\t" \
  "v_mfma_f32_16x16x32_bf16 v[184:187], v[68:71], v[92:95], v[184:187]\n\t" \
  "v_mfma_f32_16x16x32_bf16 v[188:191], v[72:75], v[92:95], v[188:191]\n\t" \
  "ds_read_b128 v[92:95], %[xa0] offset:28672\n\t" \
  "v_mfma_f32_16x16x32_bf16 v[192:195], v[60:63], v[96:99], v[192:195]\n\t" \
  "v_mfma_f32_16x16x32_bf16 v[196:199], v[64:67], v[96:99], v[196:199]\n\t" \
  "ds_read_b128 v[108:111], %[xa0] offset:36864\n\t" \
  "v_mfma_f32_16x16x32_bf16 v[200:203], v[68:71], v[96:99], v[200:203]\n\t" \
  "v_mfma_f32_16x16x32_bf16 v[204:207], v[72:75], v[96:99], v[204:207]\n\t" \
  "ds_read_b128 v[96:99], %[xa0] offset:30720\n\t" \
  "v_mfma_f32_16x16x32_bf16 v[208:211], v[60:63], v[100:103], v[208:211]\n\t" \
  "v_mfma_f32_16x16x32_bf16 v[212:215], v[64:67], v[100:103], v[212:215]\n\t" \
  "v_mfma_f32_16x16x32_bf16 v[216:219], v[68:71], v[100:103], v[216:219]\n\t" \
  "v_mfma_f32_16x16x32_bf16 v[220:223], v[72:75], v[100:103], v[220:223]\n\t" \
  "ds_read_b128 v[100:103], %[xa0] offset:32768\n\t" \
  "v_mfma_f32_16x16x32_bf16 v[224:227], v[60:63], v[36:39], v[224:227]\n\t" \
  "v_mfma_f32_16x16x32_bf16 v[228:231], v[64:67], v[36:39], v[228:231]\n\t" \
  "v_mfma_f32_16x16x32_bf16 v[232:235], v[68:71], v[36:39], v[232:235]\n\t" \
  "v_mfma_f32_16x16x32_bf16 v[236:239], v[72:75], v[36:39], v[236:239]\n\t" \
  "v_mfma_f32_16x16x32_bf16 v[240:243], v[60:63], v[40:43], v[240:243]\n\t" \
  "v_mfma_f32_16x16x32_bf16 v[244:247], v[64:67], v[40:43], v[244:247]\n\t" \
  "v_mfma_f32_16x16x32_bf16 v[248:251], v[68:71], v[40:43], v[248:251]\n\t" \
  "v_mfma_f32_16x16x32_bf16 v[252:255], v[72:75], v[40:43], v[252:255]\n\t" \
  "s_waitcnt lgkmcnt(0)\n\t" \
  "v_mfma_f32_16x16x32_bf16 v[112:115], v[44:47], v[76:79], v[112:115]\n\t" \
  "v_mfma_f32_16x16x32_bf16 v[116:119], v[48:51], v[76:79], v[116:119]\n\t" \
  "ds_read_b64_tr_b16 v[60:61], %[wl0] offset:49152\n\t" \
  "v_mfma_f32_16x16x32_bf16 v[120:123], v[52:55], v[76:79], v[120:123]\n\t" \
  "ds_read_b64_tr_b16 v[62:63], %[wl0] offset:49664\n\t" \
  "v_mfma_f32_16x16x32_bf16 v[124:127], v[56:59], v[76:79], v[124:127]\n\t" \
  "ds_read_b128 v[76:79], %[xa1] offset:20480\n\t" \
  "v_mfma_f32_16x16x32_bf16 v[128:131], v[44:47], v[80:83], v[128:131]\n\t" \
  "v_mfma_f32_16x16x32_bf16 v[132:135], v[48:51], v[80:83], v[132:135]\n\t" \
  "ds_read_b64_tr_b16 v[64:65], %[wl1] offset:49152\n\t" \
  "v_mfma_f32_16x16x32_bf16 v[136:139], v[52:55], v[80:83], v[136:139]\n\t" \
  "ds_read_b64_tr_b16 v[66:67], %[wl1] offset:49664\n\t" \
  "v_mfma_f32_16x16x32_bf16 v[140:143], v[56:59], v[80:83], v[140:143]\n\t" \
  "ds_read_b128 v[80:83], %[xa1] offset:22528\n\t" \
  "v_mfma_f32_16x16x32_bf16 v[144:147], v[44:47], v[84:87], v[144:147]\n\t" \
  "v_mfma_f32_16x16x32_bf16 v[148:151], v[48:51], v[84:87], v[148:151]\n\t" \
  "ds_read_b64_tr_b16 v[68:69], %[wl2] offset:49152\n\t" \
  "v_mfma_f32_16x16x32_bf16 v[152:155], v[52:55], v[84:87], v[152:155]\n\t" \
  "ds_read_b64_tr_b16 v[70:71], %[wl2] offset:49664\n\t" \
  "v_mfma_f32_16x16x32_bf16 v[156:159], v[56:59], v[84:87], v[156:159]\n\t" \
  "ds_read_b128 v[84:87], %[xa1] offset:24576\n\t" \
  "v_mfma_f32_16x16x32_bf16 v[160:163], v[44:47], v[88:91], v[160:163]\n\t" \
  "v_mfma_f32_16x16x32_bf16 v[164:167], v[48:51], v[88:91], v[164:167]\n\t" \
  "ds_read_b64_tr_b16 v[72:73], %[wl3] offset:49152\n\t" \
  "v_mfma_f32_16x16x32_bf16 v[168:171], v[52:55], v[88:91], v[168:171]\n\t" \
  "ds_read_b64_tr_b16 v[74:75], %[wl3] offset:49664\n\t" \
  "v_mfma_f32_16x16x32_bf16 v[172:175], v[56:59], v[88:91], v[172:175]\n\t" \
  "ds_read_b128 v[88:91], %[xa1] offset:26624\n\t" \
  "v_mfma_f32_16x16x32_bf16 v[176:179], v[44:47], v[92:95], v[176:179]\n\t" \
  "v_mfma_f32_16x16x32_bf16 v[180:183], v[48:51], v[92:95], v[180:183]\n\t" \
  "ds_read_b128 v[36:39], %[xa1] offset:34816\n\t" \
  "v_mfma_f32_16x16x32_bf16 v[184:187], v[52:55], v[92:95], v[184:187]\n\t" \
  "v_mfma_f32_16x16x32_bf16 v[188:191], v[56:59], v[92:95], v[188:191]\n\t" \
  "ds_read_b128 v[92:95], %[xa1] offset:28672\n\t" \
  "v_mfma_f32_16x16x32_bf16 v[192:195], v[44:47], v[96:99], v[192:195]\n\t" \
  "v_mfma_f32_16x16x32_bf16 v[196:199], v[48:51], v[96:99], v[196:199]\n\t" \
  "ds_read_b128 v[40:43], %[xa1] offset:36864\n\t" \
  "v_mfma_f32_16x16x32_bf16 v[200:203], v[52:55], v[96:99], v[200:203]\n\t" \
  "v_mfma_f32_16x16x32_bf16 v[204:207], v[56:59], v[96:99], v[204:207]\n\t" \
  "ds_read_b128 v[96:99], %[xa1] offset:30720\n\t" \
  "v_mfma_f32_16x16x32_bf16 v[208:211], v[44:47], v[100:103], v[208:211]\n\t" \
  "v_mfma_f32_16x16x32_bf16 v[212:215], v[48:51], v[100:103], v[212:215]\n\t" \
  "v_mfma_f32_16x16x32_bf16 v[216:219], v[52:55], v[100:103], v[216:219]\n\t" \
  "v_mfma_f32_16x16x32_bf16 v[220:223], v[56:59], v[100:103], v[220:223]\n\t" \
  "ds_read_b128 v[100:103], %[xa1] offset:32768\n\t" \
  "v_mfma_f32_16x16x32_bf16 v[224:227], v[44:47], v[104:107], v[224:227]\n\t" \
  "v_mfma_f32_16x16x32_bf16 v[228:231], v[48:51], v[104:107], v[228:231]\n\t" \
  "v_mfma_f32_16x16x32_bf16 v[232:235], v[52:55], v[104:107], v[232:235]\n\t" \
  "v_mfma_f32_16x16x32_bf16 v[236:239], v[56:59], v[104:107], v[236:239]\n\t" \
  "v_mfma_f32_16x16x32_bf16 v[240:243], v[44:47], v[108:111], v[240:243]\n\t" \
  "v_mfma_f32_16x16x32_bf16 v[244:247], v[48:51], v[108:111], v[244:247]\n\t" \
  "v_mfma_f32_16x16x32_bf16 v[248:251], v[52:55], v[108:111], v[248:251]\n\t" \
  "v_mfma_f32_16x16x32_bf16 v[252:255], v[56:59], v[108:111], v[252:255]\n\t" \
  "s_waitcnt lgkmcnt(0)\n\t" \
  "v_mfma_f32_16x16x32_bf16 v[112:115], v[60:63], v[76:79], v[112:115]\n\t" \
  "s_barrier\n\t" \
  "v_mfma_f32_16x16x32_bf16 v[116:119], v[64:67], v[76:79], v[116:119]\n\t" \
  "ds_read_b64_tr_b16 v[44:45], %[wh0] offset:0\n\t" \
  "v_mfma_f32_16x16x32_bf16 v[120:123], v[68:71], v[76:79], v[120:123]\n\t" \
  "ds_read_b64_tr_b16 v[46:47], %[wh0] offset:512\n\t" \
  "v_mfma_f32_16x16x32_bf16 v[124:127], v[72:75], v[76:79], v[124:127]\n\t" \
  "ds_read_b128 v[76:79], %[xa0] offset:40960\n\t" \
  "v_mfma_f32_16x16x32_bf16 v[128:131], v[60:63], v[80:83], v[128:131]\n\t" \
  "v_mfma_f32_16x16x32_bf16 v[132:135], v[64:67], v[80:83], v[132:135]\n\t" \
  "ds_read_b64_tr_b16 v[48:49], %[wh1] offset:0\n\t" \
  "v_mfma_f32_16x16x32_bf16 v[136:139], v[68:71], v[80:83], v[136:139]\n\t" \
  "ds_read_b64_tr_b16 v[50:51], %[wh1] offset:512\n\t" \
  "v_mfma_f32_16x16x32_bf16 v[140:143], v[72:75], v[80:83], v[140:143]\n\t" \
  "ds_read_b128 v[80:83], %[xa0] offset:43008\n\t" \
  "v_mfma_f32_16x16x32_bf16 v[144:147], v[60:63], v[84:87], v[144:147]\n\t" \
  "v_mfma_f32_16x16x32_bf16 v[148:151], v[64:67], v[84:87], v[148:151]\n\t" \
  "ds_read_b64_tr_b16 v[52:53], %[wh2] offset:0\n\t" \
  "v_mfma_f32_16x16x32_bf16 v[152:155], v[68:71], v[84:87], v[152:155]\n\t" \
  "ds_read_b64_tr_b16 v[54:55], %[wh2] offset:512\n\t" \
  "v_mfma_f32_16x16x32_bf16 v[156:159], v[72:75], v[84:87], v[156:159]\n\t" \
  "ds_read_b128 v[84:87], %[xa0] offset:45056\n\t" \
  "v_mfma_f32_16x16x32_bf16 v[160:163], v[60:63], v[88:91], v[160:163]\n\t" \
  "v_mfma_f32_16x16x32_bf16 v[164:167], v[64:67], v[88:91], v[164:167]\n\t" \
  "ds_read_b64_tr_b16 v[56:57], %[wh3] offset:0\n\t" \
  "v_mfma_f32_16x16x32_bf16 v[168:171], v[68:71], v[88:91], v[168:171]\n\t" \
  "ds_read_b64_tr_b16 v[58:59], %[wh3] offset:512\n\t" \
  "v_mfma_f32_16x16x32_bf16 v[172:175], v[72:75], v[88:91], v[172:175]\n\t" \
  "ds_read_b128 v[88:91], %[xa0] offset:47104\n\t" \
  "v_mfma_f32_16x16x32_bf16 v[176:179], v[60:63], v[92:95], v[176:179]\n\t" \
  "v_mfma_f32_16x16x32_bf16 v[180:183], v[64:67], v[92:95], v[180:183]\n\t" \
  "ds_read_b128 v[104:107], %[xa0] offset:55296\n\t" \
  "v_mfma_f32_16x16x32_bf16 v[184:187], v[68:71], v[92:95], v[184:187]\n\t" \
  "v_mfma_f32_16x16x32_bf16 v[188:191], v[72:75], v[92:95], v[188:191]\n\t" \
  "ds_read_b128 v[92:95], %[xa0] offset:49152\n\t" \
  "v_mfma_f32_16x16x32_bf16 v[192:195], v[60:63], v[96:99], v[192:195]\n\t" \
  "v_mfma_f32_16x16x32_bf16 v[196:199], v[64:67], v[96:99], v[196:199]\n\t" \
  "ds_read_b128 v[108:111], %[xa0] offset:57344\n\t" \
  "v_mfma_f32_16x16x32_bf16 v[200:203], v[68:71], v[96:99], v[200:203]\n\t" \
  "v_mfma_f32_16x16x32_bf16 v[204:207], v[72:75], v[96:99], v[204:207]\n\t" \
  "ds_read_b128 v[96:99], %[xa0] offset:51200\n\t" \
  "v_mfma_f32_16x16x32_bf16 v[208:211], v[60:63], v[100:103], v[208:211]\n\t" \
  "v_mfma_f32_16x16x32_bf16 v[212:215], v[64:67], v[100:103], v[212:215]\n\t" \
  "v_mfma_f32_16x16x32_bf16 v[216:219], v[68:71], v[100:103], v[216:219]\n\t" \
  "v_mfma_f32_16x16x32_bf16 v[220:223], v[72:75], v[100:103], v[220:223]\n\t" \
  "ds_read_b128 v[100:103], %[xa0] offset:53248\n\t" \
  "v_mfma_f32_16x16x32_bf16 v[224:227], v[60:63], v[36:39], v[224:227]\n\t" \
  "v_mfma_f32_16x16x32_bf16 v[228:231], v[64:67], v[36:39], v[228:231]\n\t" \
  "v_mfma_f32_16x16x32_bf16 v[232:235], v[68:71], v[36:39], v[232:235]\n\t" \
  "v_mfma_f32_16x16x32_bf16 v[236:239], v[72:75], v[36:39], v[236:239]\n\t" \
  "v_mfma_f32_16x16x32_bf16 v[240:243], v[60:63], v[40:43], v[240:243]\n\t" \
  "v_mfma_f32_16x16x32_bf16 v[244:247], v[64:67], v[40:43], v[244:247]\n\t" \
  "v_mfma_f32_16x16x32_bf16 v[248:251], v[68:71], v[40:43], v[248:251]\n\t" \
  "v_mfma_f32_16x16x32_bf16 v[252:255], v[72:75], v[40:43], v[252:255]\n\t" \
  "s_waitcnt lgkmcnt(0)\n\t" \
  "v_mfma_f32_16x16x32_bf16 v[112:115], v[44:47], v[76:79], v[112:115]\n\t" \
  "v_mfma_f32_16x16x32_bf16 v[116:119], v[48:51], v[76:79], v[116:119]\n\t" \
  "ds_read_b64_tr_b16 v[60:61], %[wh0] offset:16384\n\t" \
  "v_mfma_f32_16x16x32_bf16 v[120:123], v[52:55], v[76:79], v[120:123]\n\t" \
  "ds_read_b64_tr_b16 v[62:63], %[wh0] offset:16896\n\t" \
  "v_mfma_f32_16x16x32_bf16 v[124:127], v[56:59], v[76:79], v[124:127]\n\t" \
  "ds_read_b128 v[76:79], %[xa1] offset:40960\n\t" \
  "v_mfma_f32_16x16x32_bf16 v[128:131], v[44:47], v[80:83], v[128:131]\n\t" \
  "v_mfma_f32_16x16x32_bf16 v[132:135], v[48:51], v[80:83], v[132:135]\n\t" \
  "ds_read_b64_tr_b16 v[64:65], %[wh1] offset:16384\n\t" \
  "v_mfma_f32_16x16x32_bf16 v[136:139], v[52:55], v[80:83], v[136:139]\n\t" \
  "ds_read_b64_tr_b16 v[66:67], %[wh1] offset:16896\n\t" \
  "v_mfma_f32_16x16x32_bf16 v[140:143], v[56:59], v[80:83], v[140:143]\n\t" \
  "ds_read_b128 v[80:83], %[xa1] offset:43008\n\t" \
  "v_mfma_f32_16x16x32_bf16 v[144:147], v[44:47], v[84:87], v[144:147]\n\t" \
  "v_mfma_f32_16x16x32_bf16 v[148:151], v[48:51], v[84:87], v[148:151]\n\t" \
  "ds_read_b64_tr_b16 v[68:69], %[wh2] offset:16384\n\t" \
  "v_mfma_f32_16x16x32_bf16 v[152:155], v[52:55], v[84:87], v[152:155]\n\t" \
  "ds_read_b64_tr_b16 v[70:71], %[wh2] offset:16896\n\t" \
  "v_mfma_f32_16x16x32_bf16 v[156:159], v[56:59], v[84:87], v[156:159]\n\t" \
  "ds_read_b128 v[84:87], %[xa1] offset:45056\n\t" \
  "v_mfma_f32_16x16x32_bf16 v[160:163], v[44:47], v[88:91], v[160:163]\n\t" \
  "v_mfma_f32_16x16x32_bf16 v[164:167], v[48:51], v[88:91], v[164:167]\n\t" \
  "ds_read_b64_tr_b16 v[72:73], %[wh3] offset:16384\n\t" \
  "v_mfma_f32_16x16x32_bf16 v[168:171], v[52:55], v[88:91], v[168:171]\n\t" \
  "ds_read_b64_tr_b16 v[74:75], %[wh3] offset:16896\n\t" \
  "v_mfma_f32_16x16x32_bf16 v[172:175], v[56:59], v[88:91], v[172:175]\n\t" \
  "ds_read_b128 v[88:91], %[xa1] offset:47104\n\t" \
  "v_mfma_f32_16x16x32_bf16 v[176:179], v[44:47], v[92:95], v[176:179]\n\t" \
  "v_mfma_f32_16x16x32_bf16 v[180:183], v[48:51], v[92:95], v[180:183]\n\t" \
  "ds_read_b128 v[36:39], %[xa1] offset:55296\n\t" \
  "v_mfma_f32_16x16x32_bf16 v[184:187], v[52:55], v[92:95], v[184:187]\n\t" \
  "v_mfma_f32_16x16x32_bf16 v[188:191], v[56:59], v[92:95], v[188:191]\n\t" \
  "ds_read_b128 v[92:95], %[xa1] offset:49152\n\t" \
  "v_mfma_f32_16x16x32_bf16 v[192:195], v[44:47], v[96:99], v[192:195]\n\t" \
  "v_mfma_f32_16x16x32_bf16 v[196:199], v[48:51], v[96:99], v[196:199]\n\t" \
  "ds_read_b128 v[40:43], %[xa1] offset:57344\n\t" \
  "v_mfma_f32_16x16x32_bf16 v[200:203], v[52:55], v[96:99], v[200:203]\n\t" \
  "v_mfma_f32_16x16x32_bf16 v[204:207], v[56:59], v[96:99], v[204:207]\n\t" \
  "ds_read_b128 v[96:99], %[xa1] offset:51200\n\t" \
  "v_mfma_f32_16x16x32_bf16 v[208:211], v[44:47], v[100:103], v[208:211]\n\t" \
  "v_mfma_f32_16x16x32_bf16 v[212:215], v[48:51], v[100:103], v[212:215]\n\t" \
  "v_mfma_f32_16x16x32_bf16 v[216:219], v[52:55], v[100:103], v[216:219]\n\t" \
  "v_mfma_f32_16x16x32_bf16 v[220:223], v[56:59], v[100:103], v[220:223]\n\t" \
  "ds_read_b128 v[100:103], %[xa1] offset:53248\n\t" \
  "v_mfma_f32_16x16x32_bf16 v[224:227], v[44:47], v[104:107], v[224:227]\n\t" \
  "v_mfma_f32_16x16x32_bf16 v[228:231], v[48:51], v[104:107], v[228:231]\n\t" \
  "v_mfma_f32_16x16x32_bf16 v[232:235], v[52:55], v[104:107], v[232:235]\n\t" \
  "v_mfma_f32_16x16x32_bf16 v[236:239], v[56:59], v[104:107], v[236:239]\n\t" \
  "v_mfma_f32_16x16x32_bf16 v[240:243], v[44:47], v[108:111], v[240:243]\n\t" \
  "v_mfma_f32_16x16x32_bf16 v[244:247], v[48:51], v[108:111], v[244:247]\n\t" \
  "v_mfma_f32_16x16x32_bf16 v[248:251], v[52:55], v[108:111], v[248:251]\n\t" \
  "v_mfma_f32_16x16x32_bf16 v[252:255], v[56:59], v[108:111], v[252:255]\n\t" \
  "s_waitcnt lgkmcnt(0)\n\t" \
  "v_mfma_f32_16x16x32_bf16 v[112:115], v[60:63], v[76:79], v[112:115]\n\t" \
  "s_barrier\n\t" \
  "v_mfma_f32_16x16x32_bf16 v[116:119], v[64:67], v[76:79], v[116:119]\n\t" \
  "ds_read_b64_tr_b16 v[44:45], %[wl0] offset:0\n\t" \
  "v_mfma_f32_16x16x32_bf16 v[120:123], v[68:71], v[76:79], v[120:123]\n\t" \
  "ds_read_b64_tr_b16 v[46:47], %[wl0] offset:512\n\t" \
  "v_mfma_f32_16x16x32_bf16 v[124:127], v[72:75], v[76:79], v[124:127]\n\t" \
  "ds_read_b128 v[76:79], %[xa0] offset:0\n\t" \
  "v_mfma_f32_16x16x32_bf16 v[128:131], v[60:63], v[80:83], v[128:131]\n\t" \
  "v_mfma_f32_16x16x32_bf16 v[132:135], v[64:67], v[80:83], v[132:135]\n\t" \
  "ds_read_b64_tr_b16 v[48:49], %[wl1] offset:0\n\t" \
  "v_mfma_f32_16x16x32_bf16 v[136:139], v[68:71], v[80:83], v[136:139]\n\t" \
  "ds_read_b64_tr_b16 v[50:51], %[wl1] offset:512\n\t" \
  "v_mfma_f32_16x16x32_bf16 v[140:143], v[72:75], v[80:83], v[140:143]\n\t" \
  "ds_read_b128 v[80:83], %[xa0] offset:2048\n\t" \
  "v_mfma_f32_16x16x32_bf16 v[144:147], v[60:63], v[84:87], v[144:147]\n\t" \
  "v_mfma_f32_16x16x32_bf16 v[148:151], v[64:67], v[84:87], v[148:151]\n\t" \
  "ds_read_b64_tr_b16 v[52:53], %[wl2] offset:0\n\t" \
  "v_mfma_f32_16x16x32_bf16 v[152:155], v[68:71], v[84:87], v[152:155]\n\t" \
  "ds_read_b64_tr_b16 v[54:55], %[wl2] offset:512\n\t" \
  "v_mfma_f32_16x16x32_bf16 v[156:159], v[72:75], v[84:87], v[156:159]\n\t" \
  "ds_read_b128 v[84:87], %[xa0] offset:4096\n\t" \
  "v_mfma_f32_16x16x32_bf16 v[160:163], v[60:63], v[88:91], v[160:163]\n\t" \
  "v_mfma_f32_16x16x32_bf16 v[164:167], v[64:67], v[88:91], v[164:167]\n\t" \
  "ds_read_b64_tr_b16 v[56:57], %[wl3] offset:0\n\t" \
  "v_mfma_f32_16x16x32_bf16 v[168:171], v[68:71], v[88:91], v[168:171]\n\t" \
  "ds_read_b64_tr_b16 v[58:59], %[wl3] offset:512\n\t" \
  "v_mfma_f32_16x16x32_bf16 v[172:175], v[72:75], v[88:91], v[172:175]\n\t" \
  "ds_read_b128 v[88:91], %[xa0] offset:6144\n\t" \
  "v_mfma_f32_16x16x32_bf16 v[176:179], v[60:63], v[92:95], v[176:179]\n\t" \
  "v_mfma_f32_16x16x32_bf16 v[180:183], v[64:67], v[92:95], v[180:183]\n\t" \
  "ds_read_b128 v[104:107], %[xa0] offset:14336\n\t" \
  "v_mfma_f32_16x16x32_bf16 v[184:187], v[68:71], v[92:95], v[184:187]\n\t" \
  "v_mfma_f32_16x16x32_bf16 v[188:191], v[72:75], v[92:95], v[188:191]\n\t" \
  "ds_read_b128 v[92:95], %[xa0] offset:8192\n\t" \
  "v_mfma_f32_16x16x32_bf16 v[192:195], v[60:63], v[96:99], v[192:195]\n\t" \
  "v_mfma_f32_16x16x32_bf16 v[196:199], v[64:67], v[96:99], v[196:199]\n\t" \
  "ds_read_b128 v[108:111], %[xa0] offset:16384\n\t" \
  "v_mfma_f32_16x16x32_bf16 v[200:203], v[68:71], v[96:99], v[200:203]\n\t" \
  "v_mfma_f32_16x16x32_bf16 v[204:207], v[72:75], v[96:99], v[204:207]\n\t" \
  "ds_read_b128 v[96:99], %[xa0] offset:10240\n\t" \
  "v_mfma_f32_16x16x32_bf16 v[208:211], v[60:63], v[100:103], v[208:211]\n\t" \
  "v_mfma_f32_16x16x32_bf16 v[212:215], v[64:67], v[100:103], v[212:215]\n\t" \
  "v_mfma_f32_16x16x32_bf16 v[216:219], v[68:71], v[100:103], v[216:219]\n\t" \
  "v_mfma_f32_16x16x32_bf16 v[220:223], v[72:75], v[100:103], v[220:223]\n\t" \
  "ds_read_b128 v[100:103], %[xa0] offset:12288\n\t" \
  "v_mfma_f32_16x16x32_bf16 v[224:227], v[60:63], v[36:39], v[224:227]\n\t" \
  "v_mfma_f32_16x16x32_bf16 v[228:231], v[64:67], v[36:39], v[228:231]\n\t" \
  "v_mfma_f32_16x16x32_bf16 v[232:235], v[68:71], v[36:39], v[232:235]\n\t" \
  "s_sub_u32 %[n], %[n], 1\n\t" \
  "v_mfma_f32_16x16x32_bf16 v[236:239], v[72:75], v[36:39], v[236:239]\n\t" \
  "s_cmp_lg_u32 %[n], 0\n\t" \
  "v_mfma_f32_16x16x32_bf16 v[240:243], v[60:63], v[40:43], v[240:243]\n\t" \
  "v_mfma_f32_16x16x32_bf16 v[244:247], v[64:67], v[40:43], v[244:247]\n\t" \
  "v_mfma_f32_16x16x32_bf16 v[248:251], v[68:71], v[40:43], v[248:251]\n\t" \
  "v_mfma_f32_16x16x32_bf16 v[252:255], v[72:75], v[40:43], v[252:255]\n\t" \
  "s_cbranch_scc1 Lloop_%=\n\t" \
  "s_waitcnt lgkmcnt(0)\n\t" \
  "v_mfma_f32_16x16x32_bf16 v[112:115], v[44:47], v[76:79], v[112:115]\n\t" \
  "v_mfma_f32_16x16x32_bf16 v[116:119], v[48:51], v[76:79], v[116:119]\n\t" \
  "ds_read_b64_tr_b16 v[60:61], %[wl0] offset:16384\n\t" \
  "v_mfma_f32_16x16x32_bf16 v[120:123], v[52:55], v[76:79], v[120:123]\n\t" \
  "ds_read_b64_tr_b16 v[62:63], %[wl0] offset:16896\n\t" \
  "v_mfma_f32_16x16x32_bf16 v[124:127], v[56:59], v[76:79], v[124:127]\n\t" \
  "ds_read_b128 v[76:79], %[xa1] offset:0\n\t" \
  "v_mfma_f32_16x16x32_bf16 v[128:131], v[44:47], v[80:83], v[128:131]\n\t" \
  "v_mfma_f32_16x16x32_bf16 v[132:135], v[48:51], v[80:83], v[132:135]\n\t" \
  "ds_read_b64_tr_b16 v[64:65], %[wl1] offset:16384\n\t" \
  "v_mfma_f32_16x16x32_bf16 v[136:139], v[52:55], v[80:83], v[136:139]\n\t" \
  "ds_read_b64_tr_b16 v[66:67], %[wl1] offset:16896\n\t" \
  "v_mfma_f32_16x16x32_bf16 v[140:143], v[56:59], v[80:83], v[140:143]\n\t" \
  "ds_read_b128 v[80:83], %[xa1] offset:2048\n\t" \
  "v_mfma_f32_16x16x32_bf16 v[144:147], v[44:47], v[84:87], v[144:147]\n\t" \
  "v_mfma_f32_16x16x32_bf16 v[148:151], v[48:51], v[84:87], v[148:151]\n\t" \
  "ds_read_b64_tr_b16 v[68:69], %[wl2] offset:16384\n\t" \
  "v_mfma_f32_16x16x32_bf16 v[152:155], v[52:55], v[84:87], v[152:155]\n\t" \
  "ds_read_b64_tr_b16 v[70:71], %[wl2] offset:16896\n\t" \
  "v_mfma_f32_16x16x32_bf16 v[156:159], v[56:59], v[84:87], v[156:159]\n\t" \
  "ds_read_b128 v[84:87], %[xa1] offset:4096\n\t" \
  "v_mfma_f32_16x16x32_bf16 v[160:163], v[44:47], v[88:91], v[160:163]\n\t" \
  "v_mfma_f32_16x16x32_bf16 v[164:167], v[48:51], v[88:91], v[164:167]\n\t" \
  "ds_read_b64_tr_b16 v[72:73], %[wl3] offset:16384\n\t" \
  "v_mfma_f32_16x16x32_bf16 v[168:171], v[52:55], v[88:91], v[168:171]\n\t" \
  "ds_read_b64_tr_b16 v[74:75], %[wl3] offset:16896\n\t" \
  "v_mfma_f32_16x16x32_bf16 v[172:175], v[56:59], v[88:91], v[172:175]\n\t" \
  "ds_read_b128 v[88:91], %[xa1] offset:6144\n\t" \
  "v_mfma_f32_16x16x32_bf16 v[176:179], v[44:47], v[92:95], v[176:179]\n\t" \
  "v_mfma_f32_16x16x32_bf16 v[180:183], v[48:51], v[92:95], v[180:183]\n\t" \
  "ds_read_b128 v[36:39], %[xa1] offset:14336\n\t" \
  "v_mfma_f32_16x16x32_bf16 v[184:187], v[52:55], v[92:95], v[184:187]\n\t" \
  "v_mfma_f32_16x16x32_bf16 v[188:191], v[56:59], v[92:95], v[188:191]\n\t" \
  "ds_read_b128 v[92:95], %[xa1] offset:8192\n\t" \
  "v_mfma_f32_16x16x32_bf16 v[192:195], v[44:47], v[96:99], v[192:195]\n\t" \
  "v_mfma_f32_16x16x32_bf16 v[196:199], v[48:51], v[96:99], v[196:199]\n\t" \
  "ds_read_b128 v[40:43], %[xa1] offset:16384\n\t" \
  "v_mfma_f32_16x16x32_bf16 v[200:203], v[52:55], v[96:99], v[200:203]\n\t" \
  "v_mfma_f32_16x16x32_bf16 v[204:207], v[56:59], v[96:99], v[204:207]\n\t" \
  "ds_read_b128 v[96:99], %[xa1] offset:10240\n\t" \
  "v_mfma_f32_16x16x32_bf16 v[208:211], v[44:47], v[100:103], v[208:211]\n\t" \
  "v_mfma_f32_16x16x32_bf16 v[212:215], v[48:51], v[100:103], v[212:215]\n\t" \
  "v_mfma_f32_16x16x32_bf16 v[216:219], v[52:55], v[100:103], v[216:219]\n\t" \
  "v_mfma_f32_16x16x32_bf16 v[220:223], v[56:59], v[100:103], v[220:223]\n\t" \
  "ds_read_b128 v[100:103], %[xa1] offset:12288\n\t" \
  "ds_read_b32 %[bd], %[bd]\n\t" \
  "v_mfma_f32_16x16x32_bf16 v[224:227], v[44:47], v[104:107], v[224:227]\n\t" \
  "v_mfma_f32_16x16x32_bf16 v[228:231], v[48:51], v[104:107], v[228:231]\n\t" \
  "v_mfma_f32_16x16x32_bf16 v[232:235], v[52:55], v[104:107], v[232:235]\n\t" \
  "v_mfma_f32_16x16x32_bf16 v[236:239], v[56:59], v[104:107], v[236:239]\n\t" \
  "v_mfma_f32_16x16x32_bf16 v[240:243], v[44:47], v[108:111], v[240:243]\n\t" \
  "v_mfma_f32_16x16x32_bf16 v[244:247], v[48:51], v[108:111], v[244:247]\n\t" \
  "v_mfma_f32_16x16x32_bf16 v[248:251], v[52:55], v[108:111], v[248:251]\n\t" \
  "v_mfma_f32_16x16x32_bf16 v[252:255], v[56:59], v[108:111], v[252:255]\n\t" \
  "s_waitcnt lgkmcnt(0)\n\t" \
  "s_waitcnt vmcnt(0)\n\t" \
  "s_nop 7\n\t" \
  "s_nop 7\n\t" \
  "s_nop 7\n\t" \
  ""
#define CONS_ASM9X \
  "buffer_load_dword %[pd], %[pv], %[rw], %[po] offen\n\t" \
  "s_add_u32 %[po], %[po], %[pi]\n\t" \
  "buffer_load_dword %[pd], %[pv], %[rw], %[po] offen\n\t" \
  "s_add_u32 %[po], %[po], %[pi]\n\t" \
  "v_mov_b32 v112, 0\n\t" \
  "v_mov_b32 v113, 0\n\t" \
  "v_mov_b32 v114, 0\n\t" \
  "v_mov_b32 v115, 0\n\t" \
  "v_mov_b32 v116, 0\n\t" \
  "v_mov_b32 v117, 0\n\t" \
  "v_mov_b32 v118, 0\n\t" \
  "v_mov_b32 v119, 0\n\t" \
  "v_mov_b32 v120, 0\n\t" \
  "v_mov_b32 v121, 0\n\t" \
  "v_mov_b32 v122, 0\n\t" \
  "v_mov_b32 v123, 0\n\t" \
  "v_mov_b32 v124, 0\n\t" \
  "v_mov_b32 v125, 0\n\t" \
  "v_mov_b32 v126, 0\n\t" \
  "v_mov_b32 v127, 0\n\t" \
  "v_mov_b32 v128, 0\n\t" \
  "v_mov_b32 v129, 0\n\t" \
  "v_mov_b32 v130, 0\n\t" \
  "v_mov_b32 v131, 0\n\t" \
  "v_mov_b32 v132, 0\n\t" \
  "v_mov_b32 v133, 0\n\t" \
  "v_mov_b32 v134, 0\n\t" \
  "v_mov_b32 v135, 0\n\t" \
  "v_mov_b32 v136, 0\n\t" \
  "v_mov_b32 v137, 0\n\t" \
  "v_mov_b32 v138, 0\n\t" \
  "v_mov_b32 v139, 0\n\t" \
  "v_mov_b32 v140, 0\n\t" \
  "v_mov_b32 v141, 0\n\t" \
  "v_mov_b32 v142, 0\n\t" \
  "v_mov_b32 v143, 0\n\t" \
  "v_mov_b32 v144, 0\n\t" \
  "v_mov_b32 v145, 0\n\t" \
  "v_mov_b32 v146, 0\n\t" \
  "v_mov_b32 v147, 0\n\t" \
  "v_mov_b32 v148, 0\n\t" \
  "v_mov_b32 v149, 0\n\t" \
  "v_mov_b32 v150, 0\n\t" \
  "v_mov_b32 v151, 0\n\t" \
  "v_mov_b32 v152, 0\n\t" \
  "v_mov_b32 v153, 0\n\t" \
  "v_mov_b32 v154, 0\n\t" \
  "v_mov_b32 v155, 0\n\t" \
  "v_mov_b32 v156, 0\n\t" \
  "v_mov_b32 v157, 0\n\t" \
  "v_mov_b32 v158, 0\n\t" \
  "v_mov_b32 v159, 0\n\t" \
  "v_mov_b32 v160, 0\n\t" \
  "v_mov_b32 v161, 0\n\t" \
  "v_mov_b32 v162, 0\n\t" \
  "v_mov_b32 v163, 0\n\t" \
  "v_mov_b32 v164, 0\n\t" \
  "v_mov_b32 v165, 0\n\t" \
  "v_mov_b32 v166, 0\n\t" \
  "v_mov_b32 v167, 0\n\t" \
  "v_mov_b32 v168, 0\n\t" \
  "v_mov_b32 v169, 0\n\t" \
  "v_mov_b32 v170, 0\n\t" \
  "v_mov_b32 v171, 0\n\t" \
  "v_mov_b32 v172, 0\n\t" \
  "v_mov_b32 v173, 0\n\t" \
  "v_mov_b32 v174, 0\n\t" \
  "v_mov_b32 v175, 0\n\t" \
  "v_mov_b32 v176, 0\n\t" \
  "v_mov_b32 v177, 0\n\t" \
  "v_mov_b32 v178, 0\n\t" \
  "v_mov_b32 v179, 0\n\t" \
  "v_mov_b32 v180, 0\n\t" \
  "v_mov_b32 v181, 0\n\t" \
  "v_mov_b32 v182, 0\n\t" \
  "v_mov_b32 v183, 0\n\t" \
  "v_mov_b32 v184, 0\n\t" \
  "v_mov_b32 v185, 0\n\t" \
  "v_mov_b32 v186, 0\n\t" \
  "v_mov_b32 v187, 0\n\t" \
  "v_mov_b32 v188, 0\n\t" \
  "v_mov_b32 v189, 0\n\t" \
  "v_mov_b32 v190, 0\n\t" \
  "v_mov_b32 v191, 0\n\t" \
  "v_mov_b32 v192, 0\n\t" \
  "v_mov_b32 v193, 0\n\t" \
  "v_mov_b32 v194, 0\n\t" \
  "v_mov_b32 v195, 0\n\t" \
  "v_mov_b32 v196, 0\n\t" \
  "v_mov_b32 v197, 0\n\t" \
  "v_mov_b32 v198, 0\n\t" \
  "v_mov_b32 v199, 0\n\t" \
  "v_mov_b32 v200, 0\n\t" \
  "v_mov_b32 v201, 0\n\t" \
  "v_mov_b32 v202, 0\n\t" \
  "v_mov_b32 v203, 0\n\t" \
  "v_mov_b32 v204, 0\n\t" \
  "v_mov_b32 v205, 0\n\t" \
  "v_mov_b32 v206, 0\n\t" \
  "v_mov_b32 v207, 0\n\t" \
  "v_mov_b32 v208, 0\n\t" \
  "v_mov_b32 v209, 0\n\t" \
  "v_mov_b32 v210, 0\n\t" \
  "v_mov_b32 v211, 0\n\t" \
  "v_mov_b32 v212, 0\n\t" \
  "v_mov_b32 v213, 0\n\t" \
  "v_mov_b32 v214, 0\n\t" \
  "v_mov_b32 v215, 0\n\t" \
  "v_mov_b32 v216, 0\n\t" \
  "v_mov_b32 v217, 0\n\t" \
  "v_mov_b32 v218, 0\n\t" \
  "v_mov_b32 v219, 0\n\t" \
  "v_mov_b32 v220, 0\n\t" \
  "v_mov_b32 v221, 0\n\t" \
  "v_mov_b32 v222, 0\n\t" \
  "v_mov_b32 v223, 0\n\t" \
  "v_mov_b32 v224, 0\n\t" \
  "v_mov_b32 v225, 0\n\t" \
  "v_mov_b32 v226, 0\n\t" \
  "v_mov_b32 v227, 0\n\t" \
  "v_mov_b32 v228, 0\n\t" \
  "v_mov_b32 v229, 0\n\t" \
  "v_mov_b32 v230, 0\n\t" \
  "v_mov_b32 v231, 0\n\t" \
  "v_mov_b32 v232, 0\n\t" \
  "v_mov_b32 v233, 0\n\t" \
  "v_mov_b32 v234, 0\n\t" \
  "v_mov_b32 v235, 0\n\t" \
  "v_mov_b32 v236, 0\n\t" \
  "v_mov_b32 v237, 0\n\t" \
  "v_mov_b32 v238, 0\n\t" \
  "v_mov_b32 v239, 0\n\t" \
  "v_mov_b32 v240, 0\n\t" \
  "v_mov_b32 v241, 0\n\t" \
  "v_mov_b32 v242, 0\n\t" \
  "v_mov_b32 v243, 0\n\t" \
  "v_mov_b32 v244, 0\n\t" \
  "v_mov_b32 v245, 0\n\t" \
  "v_mov_b32 v246, 0\n\t" \
  "v_mov_b32 v247, 0\n\t" \
  "v_mov_b32 v248, 0\n\t" \
  "v_mov_b32 v249, 0\n\t" \
  "v_mov_b32 v250, 0\n\t" \
  "v_mov_b32 v251, 0\n\t" \
  "v_mov_b32 v252, 0\n\t" \
  "v_mov_b32 v253, 0\n\t" \
  "v_mov_b32 v254, 0\n\t" \
  "v_mov_b32 v255, 0\n\t" \
  "v_mov_b32 v28, 0\n\t" \
  "v_mov_b32 v29, 0\n\t" \
  "v_mov_b32 v30, 0\n\t" \
  "v_mov_b32 v31, 0\n\t" \
  "s_barrier\n\t" \
  "ds_read_b64_tr_b16 v[44:45], %[wl0] offset:0\n\t" \
  "ds_read_b64_tr_b16 v[46:47], %[wl0] offset:512\n\t" \
  "ds_read_b64_tr_b16 v[48:49], %[wl1] offset:0\n\t" \
  "ds_read_b64_tr_b16 v[50:51], %[wl1] offset:512\n\t" \
  "ds_read_b64_tr_b16 v[52:53], %[wl2] offset:0\n\t" \
  "ds_read_b64_tr_b16 v[54:55], %[wl2] offset:512\n\t" \
  "ds_read_b64_tr_b16 v[56:57], %[wl3] offset:0\n\t" \
  "ds_read_b64_tr_b16 v[58:59], %[wl3] offset:512\n\t" \
  "ds_read_b128 v[76:79], %[xa0] offset:0\n\t" \
  "ds_read_b128 v[80:83], %[xa0] offset:2048\n\t" \
  "ds_read_b128 v[84:87], %[xa0] offset:4096\n\t" \
  "ds_read_b128 v[88:91], %[xa0] offset:6144\n\t" \
  "ds_read_b128 v[92:95], %[xa0] offset:8192\n\t" \
  "ds_read_b128 v[96:99], %[xa0] offset:10240\n\t" \
  "ds_read_b128 v[100:103], %[xa0] offset:12288\n\t" \
  "ds_read_b128 v[104:107], %[xa0] offset:14336\n\t" \
  "ds_read_b128 v[108:111], %[xa0] offset:16384\n\t" \
  "ds_read_b128 v[32:35], %[xa0] offset:18432\n\t" \
  "Lloop_%=:\n\t" \
  "s_waitcnt lgkmcnt(0)\n\t" \
  "v_mfma_f32_16x16x32_bf16 v[28:31], v[44:47], v[32:35], v[28:31]\n\t" \
  "v_mfma_f32_16x16x32_bf16 v[112:115], v[44:47], v[76:79], v[112:115]\n\t" \
  "v_mfma_f32_16x16x32_bf16 v[116:119], v[48:51], v[76:79], v[116:119]\n\t" \
  "ds_read_b64_tr_b16 v[60:61], %[wl0] offset:16384\n\t" \
  "v_mfma_f32_16x16x32_bf16 v[120:123], v[52:55], v[76:79], v[120:123]\n\t" \
  "ds_read_b64_tr_b16 v[62:63], %[wl0] offset:16896\n\t" \
  "v_mfma_f32_16x16x32_bf16 v[124:127], v[56:59], v[76:79], v[124:127]\n\t" \
  "ds_read_b128 v[76:79], %[xa1] offset:0\n\t" \
  "v_mfma_f32_16x16x32_bf16 v[128:131], v[44:47], v[80:83], v[128:131]\n\t" \
  "v_mfma_f32_16x16x32_bf16 v[132:135], v[48:51], v[80:83], v[132:135]\n\t" \
  "ds_read_b64_tr_b16 v[64:65], %[wl1] offset:16384\n\t" \
  "v_mfma_f32_16x16x32_bf16 v[136:139], v[52:55], v[80:83], v[136:139]\n\t" \
  "ds_read_b64_tr_b16 v[66:67], %[wl1] offset:16896\n\t" \
  "v_mfma_f32_16x16x32_bf16 v[140:143], v[56:59], v[80:83], v[140:143]\n\t" \
  "ds_read_b128 v[80:83], %[xa1] offset:2048\n\t" \
  "v_mfma_f32_16x16x32_bf16 v[144:147], v[44:47], v[84:87], v[144:147]\n\t" \
  "v_mfma_f32_16x16x32_bf16 v[148:151], v[48:51], v[84:87], v[148:151]\n\t" \
  "ds_read_b64_tr_b16 v[68:69], %[wl2] offset:16384\n\t" \
  "v_mfma_f32_16x16x32_bf16 v[152:155], v[52:55], v[84:87], v[152:155]\n\t" \
  "ds_read_b64_tr_b16 v[70:71], %[wl2] offset:16896\n\t" \
  "v_mfma_f32_16x16x32_bf16 v[156:159], v[56:59], v[84:87], v[156:159]\n\t" \
  "ds_read_b128 v[84:87], %[xa1] offset:4096\n\t" \
  "v_mfma_f32_16x16x32_bf16 v[160:163], v[44:47], v[88:91], v[160:163]\n\t" \
  "v_mfma_f32_16x16x32_bf16 v[164:167], v[48:51], v[88:91], v[164:167]\n\t" \
  "ds_read_b64_tr_b16 v[72:73], %[wl3] offset:16384\n\t" \
  "v_mfma_f32_16x16x32_bf16 v[168:171], v[52:55], v[88:91], v[168:171]\n\t" \
  "ds_read_b64_tr_b16 v[74:75], %[wl3] offset:16896\n\t" \
  "v_mfma_f32_16x16x32_bf16 v[172:175], v[56:59], v[88:91], v[172:175]\n\t" \
  "ds_read_b128 v[88:91], %[xa1] offset:6144\n\t" \
  "v_mfma_f32_16x16x32_bf16 v[176:179], v[44:47], v[92:95], v[176:179]\n\t" \
  "v_mfma_f32_16x16x32_bf16 v[180:183], v[48:51], v[92:95], v[180:183]\n\t" \
  "ds_read_b128 v[36:39], %[xa1] offset:14336\n\t" \
  "v_mfma_f32_16x16x32_bf16 v[184:187], v[52:55], v[92:95], v[184:187]\n\t" \
  "v_mfma_f32_16x16x32_bf16 v[188:191], v[56:59], v[92:95], v[188:191]\n\t" \
  "ds_read_b128 v[92:95], %[xa1] offset:8192\n\t" \
  "v_mfma_f32_16x16x32_bf16 v[192:195], v[44:47], v[96:99], v[192:195]\n\t" \
  "v_mfma_f32_16x16x32_bf16 v[196:199], v[48:51], v[96:99], v[196:199]\n\t" \
  "ds_read_b128 v[40:43], %[xa1] offset:16384\n\t" \
  "v_mfma_f32_16x16x32_bf16 v[200:203], v[52:55], v[96:99], v[200:203]\n\t" \
  "v_mfma_f32_16x16x32_bf16 v[204:207], v[56:59], v[96:99], v[204:207]\n\t" \
  "ds_read_b128 v[96:99], %[xa1] offset:10240\n\t" \
  "v_mfma_f32_16x16x32_bf16 v[208:211], v[44:47], v[100:103], v[208:211]\n\t" \
  "v_mfma_f32_16x16x32_bf16 v[212:215], v[48:51], v[100:103], v[212:215]\n\t" \
  "ds_read_b128 v[32:35], %[xa1] offset:18432\n\t" \
  "v_mfma_f32_16x16x32_bf16 v[216:219], v[52:55], v[100:103], v[216:219]\n\t" \
  "v_mfma_f32_16x16x32_bf16 v[220:223], v[56:59], v[100:103], v[220:223]\n\t" \
  "ds_read_b128 v[100:103], %[xa1] offset:12288\n\t" \
  "v_mfma_f32_16x16x32_bf16 v[224:227], v[44:47], v[104:107], v[224:227]\n\t" \
  "buffer_load_dword %[pd], %[pv], %[rw], %[po] offen\n\t" \
  "s_add_u32 %[po], %[po], %[pi]\n\t" \
  "v_mfma_f32_16x16x32_bf16 v[228:231], v[48:51], v[104:107], v[228:231]\n\t" \
  "v_mfma_f32_16x16x32_bf16 v[232:235], v[52:55], v[104:107], v[232:235]\n\t" \
  "v_mfma_f32_16x16x32_bf16 v[236:239], v[56:59], v[104:107], v[236:239]\n\t" \
  "v_mfma_f32_16x16x32_bf16 v[240:243], v[44:47], v[108:111], v[240:243]\n\t" \
  "v_mfma_f32_16x16x32_bf16 v[244:247], v[48:51], v[108:111], v[244:247]\n\t" \
  "v_mfma_f32_16x16x32_bf16 v[248:251], v[52:55], v[108:111], v[248:251]\n\t" \
  "v_mfma_f32_16x16x32_bf16 v[252:255], v[56:59], v[108:111], v[252:255]\n\t" \
  "s_waitcnt lgkmcnt(0)\n\t" \
  "v_mfma_f32_16x16x32_bf16 v[28:31], v[60:63], v[32:35], v[28:31]\n\t" \
  "s_barrier\n\t" \
  "v_mfma_f32_16x16x32_bf16 v[112:115], v[60:63], v[76:79], v[112:115]\n\t" \
  "v_mfma_f32_16x16x32_bf16 v[116:119], v[64:67], v[76:79], v[116:119]\n\t" \
  "ds_read_b64_tr_b16 v[44:45], %[wl0] offset:32768\n\t" \
  "v_mfma_f32_16x16x32_bf16 v[120:123], v[68:71], v[76:79], v[120:123]\n\t" \
  "ds_read_b64_tr_b16 v[46:47], %[wl0] offset:33280\n\t" \
  "v_mfma_f32_16x16x32_bf16 v[124:127], v[72:75], v[76:79], v[124:127]\n\t" \
  "ds_read_b128 v[76:79], %[xa0] offset:20480\n\t" \
  "v_mfma_f32_16x16x32_bf16 v[128:131], v[60:63], v[80:83], v[128:131]\n\t" \
  "v_mfma_f32_16x16x32_bf16 v[132:135], v[64:67], v[80:83], v[132:135]\n\t" \
  "ds_read_b64_tr_b16 v[48:49], %[wl1] offset:32768\n\t" \
  "v_mfma_f32_16x16x32_bf16 v[136:139], v[68:71], v[80:83], v[136:139]\n\t" \
  "ds_read_b64_tr_b16 v[50:51], %[wl1] offset:33280\n\t" \
  "v_mfma_f32_16x16x32_bf16 v[140:143], v[72:75], v[80:83], v[140:143]\n\t" \
  "ds_read_b128 v[80:83], %[xa0] offset:22528\n\t" \
  "v_mfma_f32_16x16x32_bf16 v[144:147], v[60:63], v[84:87], v[144:147]\n\t" \
  "v_mfma_f32_16x16x32_bf16 v[148:151], v[64:67], v[84:87], v[148:151]\n\t" \
  "ds_read_b64_tr_b16 v[52:53], %[wl2] offset:32768\n\t" \
  "v_mfma_f32_16x16x32_bf16 v[152:155], v[68:71], v[84:87], v[152:155]\n\t" \
  "ds_read_b64_tr_b16 v[54:55], %[wl2] offset:33280\n\t" \
  "v_mfma_f32_16x16x32_bf16 v[156:159], v[72:75], v[84:87], v[156:159]\n\t" \
  "ds_read_b128 v[84:87], %[xa0] offset:24576\n\t" \
  "v_mfma_f32_16x16x32_bf16 v[160:163], v[60:63], v[88:91], v[160:163]\n\t" \
  "v_mfma_f32_16x16x32_bf16 v[164:167], v[64:67], v[88:91], v[164:167]\n\t" \
  "ds_read_b64_tr_b16 v[56:57], %[wl3] offset:32768\n\t" \
  "v_mfma_f32_16x16x32_bf16 v[168:171], v[68:71], v[88:91], v[168:171]\n\t" \
  "ds_read_b64_tr_b16 v[58:59], %[wl3] offset:33280\n\t" \
  "v_mfma_f32_16x16x32_bf16 v[172:175], v[72:75], v[88:91], v[172:175]\n\t" \
  "ds_read_b128 v[88:91], %[xa0] offset:26624\n\t" \
  "v_mfma_f32_16x16x32_bf16 v[176:179], v[60:63], v[92:95], v[176:179]\n\t" \
  "v_mfma_f32_16x16x32_bf16 v[180:183], v[64:67], v[92:95], v[180:183]\n\t" \
  "ds_read_b128 v[104:107], %[xa0] offset:34816\n\t" \
  "v_mfma_f32_16x16x32_bf16 v[184:187], v[68:71], v[92:95], v[184:187]\n\t" \
  "v_mfma_f32_16x16x32_bf16 v[188:191], v[72:75], v[92:95], v[188:191]\n\t" \
  "ds_read_b128 v[92:95], %[xa0] offset:28672\n\t" \
  "v_mfma_f32_16x16x32_bf16 v[192:195], v[60:63], v[96:99], v[192:195]\n\t" \
  "v_mfma_f32_16x16x32_bf16 v[196:199], v[64:67], v[96:99], v[196:199]\n\t" \
  "ds_read_b128 v[108:111], %[xa0] offset:36864\n\t" \
  "v_mfma_f32_16x16x32_bf16 v[200:203], v[68:71], v[96:99], v[200:203]\n\t" \
  "v_mfma_f32_16x16x32_bf16 v[204:207], v[72:75], v[96:99], v[204:207]\n\t" \
  "ds_read_b128 v[96:99], %[xa0] offset:30720\n\t" \
  "v_mfma_f32_16x16x32_bf16 v[208:211], v[60:63], v[100:103], v[208:211]\n\t" \
  "v_mfma_f32_16x16x32_bf16 v[212:215], v[64:67], v[100:103], v[212:215]\n\t" \
  "ds_read_b128 v[32:35], %[xa0] offset:38912\n\t" \
  "v_mfma_f32_16x16x32_bf16 v[216:219], v[68:71], v[100:103], v[216:219]\n\t" \
  "v_mfma_f32_16x16x32_bf16 v[220:223], v[72:75], v[100:103], v[220:223]\n\t" \
  "ds_read_b128 v[100:103], %[xa0] offset:32768\n\t" \
  "v_mfma_f32_16x16x32_bf16 v[224:227], v[60:63], v[36:39], v[224:227]\n\t" \
  "v_mfma_f32_16x16x32_bf16 v[228:231], v[64:67], v[36:39], v[228:231]\n\t" \
  "v_mfma_f32_16x16x32_bf16 v[232:235], v[68:71], v[36:39], v[232:235]\n\t" \
  "v_mfma_f32_16x16x32_bf16 v[236:239], v[72:75], v[36:39], v[236:239]\n\t" \
  "v_mfma_f32_16x16x32_bf16 v[240:243], v[60:63], v[40:43], v[240:243]\n\t" \
  "v_mfma_f32_16x16x32_bf16 v[244:247], v[64:67], v[40:43], v[244:247]\n\t" \
  "v_mfma_f32_16x16x32_bf16 v[248:251], v[68:71], v[40:43], v[248:251]\n\t" \
  "v_mfma_f32_16x16x32_bf16 v[252:255], v[72:75], v[40:43], v[252:255]\n\t" \
  "s_waitcnt lgkmcnt(0)\n\t" \
  "v_mfma_f32_16x16x32_bf16 v[28:31], v[44:47], v[32:35], v[28:31]\n\t" \
  "v_mfma_f32_16x16x32_bf16 v[112:115], v[44:47], v[76:79], v[112:115]\n\t" \
  "v_mfma_f32_16x16x32_bf16 v[116:119], v[48:51], v[76:79], v[116:119]\n\t" \
  "ds_read_b64_tr_b16 v[60:61], %[wl0] offset:49152\n\t" \
  "v_mfma_f32_16x16x32_bf16 v[120:123], v[52:55], v[76:79], v[120:123]\n\t" \
  "ds_read_b64_tr_b16 v[62:63], %[wl0] offset:49664\n\t" \
  "v_mfma_f32_16x16x32_bf16 v[124:127], v[56:59], v[76:79], v[124:127]\n\t" \
  "ds_read_b128 v[76:79], %[xa1] offset:20480\n\t" \
  "v_mfma_f32_16x16x32_bf16 v[128:131], v[44:47], v[80:83], v[128:131]\n\t" \
  "v_mfma_f32_16x16x32_bf16 v[132:135], v[48:51], v[80:83], v[132:135]\n\t" \
  "ds_read_b64_tr_b16 v[64:65], %[wl1] offset:49152\n\t" \
  "v_mfma_f32_16x16x32_bf16 v[136:139], v[52:55], v[80:83], v[136:139]\n\t" \
  "ds_read_b64_tr_b16 v[66:67], %[wl1] offset:49664\n\t" \
  "v_mfma_f32_16x16x32_bf16 v[140:143], v[56:59], v[80:83], v[140:143]\n\t" \
  "ds_read_b128 v[80:83], %[xa1] offset:22528\n\t" \
  "v_mfma_f32_16x16x32_bf16 v[144:147], v[44:47], v[84:87], v[144:147]\n\t" \
  "v_mfma_f32_16x16x32_bf16 v[148:151], v[48:51], v[84:87], v[148:151]\n\t" \
  "ds_read_b64_tr_b16 v[68:69], %[wl2] offset:49152\n\t" \
  "v_mfma_f32_16x16x32_bf16 v[152:155], v[52:55], v[84:87], v[152:155]\n\t" \
  "ds_read_b64_tr_b16 v[70:71], %[wl2] offset:49664\n\t" \
  "v_mfma_f32_16x16x32_bf16 v[156:159], v[56:59], v[84:87], v[156:159]\n\t" \
  "ds_read_b128 v[84:87], %[xa1] offset:24576\n\t" \
  "v_mfma_f32_16x16x32_bf16 v[160:163], v[44:47], v[88:91], v[160:163]\n\t" \
  "v_mfma_f32_16x16x32_bf16 v[164:167], v[48:51], v[88:91], v[164:167]\n\t" \
  "ds_read_b64_tr_b16 v[72:73], %[wl3] offset:49152\n\t" \
  "v_mfma_f32_16x16x32_bf16 v[168:171], v[52:55], v[88:91], v[168:171]\n\t" \
  "ds_read_b64_tr_b16 v[74:75], %[wl3] offset:49664\n\t" \
  "v_mfma_f32_16x16x32_bf16 v[172:175], v[56:59], v[88:91], v[172:175]\n\t" \
  "ds_read_b128 v[88:91], %[xa1] offset:26624\n\t" \
  "v_mfma_f32_16x16x32_bf16 v[176:179], v[44:47], v[92:95], v[176:179]\n\t" \
  "v_mfma_f32_16x16x32_bf16 v[180:183], v[48:51], v[92:95], v[180:183]\n\t" \
  "ds_read_b128 v[36:39], %[xa1] offset:34816\n\t" \
  "v_mfma_f32_16x16x32_bf16 v[184:187], v[52:55], v[92:95], v[184:187]\n\t" \
  "v_mfma_f32_16x16x32_bf16 v[188:191], v[56:59], v[92:95], v[188:191]\n\t" \
  "ds_read_b128 v[92:95], %[xa1] offset:28672\n\t" \
  "v_mfma_f32_16x16x32_bf16 v[192:195], v[44:47], v[96:99], v[192:195]\n\t" \
  "v_mfma_f32_16x16x32_bf16 v[196:199], v[48:51], v[96:99], v[196:199]\n\t" \
  "ds_read_b128 v[40:43], %[xa1] offset:36864\n\t" \
  "v_mfma_f32_16x16x32_bf16 v[200:203], v[52:55], v[96:99], v[200:203]\n\t" \
  "v_mfma_f32_16x16x32_bf16 v[204:207], v[56:59], v[96:99], v[204:207]\n\t" \
  "ds_read_b128 v[96:99], %[xa1] offset:30720\n\t" \
  "v_mfma_f32_16x16x32_bf16 v[208:211], v[44:47], v[100:103], v[208:211]\n\t" \
  "v_mfma_f32_16x16x32_bf16 v[212:215], v[48:51], v[100:103], v[212:215]\n\t" \
  "ds_read_b128 v[32:35], %[xa1] offset:38912\n\t" \
  "v_mfma_f32_16x16x32_bf16 v[216:219], v[52:55], v[100:103], v[216:219]\n\t" \
  "v_mfma_f32_16x16x32_bf16 v[220:223], v[56:59], v[100:103], v[220:223]\n\t" \
  "ds_read_b128 v[100:103], %[xa1] offset:32768\n\t" \
  "v_mfma_f32_16x16x32_bf16 v[224:227], v[44:47], v[104:107], v[224:227]\n\t" \
  "v_mfma_f32_16x16x32_bf16 v[228:231], v[48:51], v[104:107], v[228:231]\n\t" \
  "v_mfma_f32_16x16x32_bf16 v[232:235], v[52:55], v[104:107], v[232:235]\n\t" \
  "v_mfma_f32_16x16x32_bf16 v[236:239], v[56:59], v[104:107], v[236:239]\n\t" \
  "v_mfma_f32_16x16x32_bf16 v[240:243], v[44:47], v[108:111], v[240:243]\n\t" \
  "v_mfma_f32_16x16x32_bf16 v[244:247], v[48:51], v[108:111], v[244:247]\n\t" \
  "v_mfma_f32_16x16x32_bf16 v[248:251], v[52:55], v[108:111], v[248:251]\n\t" \
  "v_mfma_f32_16x16x32_bf16 v[252:255], v[56:59], v[108:111], v[252:255]\n\t" \
  "s_waitcnt lgkmcnt(0)\n\t" \
  "v_mfma_f32_16x16x32_bf16 v[28:31], v[60:63], v[32:35], v[28:31]\n\t" \
  "s_barrier\n\t" \
  "v_mfma_f32_16x16x32_bf16 v[112:115], v[60:63], v[76:79], v[112:115]\n\t" \
  "v_mfma_f32_16x16x32_bf16 v[116:119], v[64:67], v[76:79], v[116:119]\n\t" \
  "ds_read_b64_tr_b16 v[44:45], %[wh0] offset:0\n\t" \
  "v_mfma_f32_16x16x32_bf16 v[120:123], v[68:71], v[76:79], v[120:123]\n\t" \
  "ds_read_b64_tr_b16 v[46:47], %[wh0] offset:512\n\t" \
  "v_mfma_f32_16x16x32_bf16 v[124:127], v[72:75], v[76:79], v[124:127]\n\t" \
  "ds_read_b128 v[76:79], %[xa0] offset:40960\n\t" \
  "v_mfma_f32_16x16x32_bf16 v[128:131], v[60:63], v[80:83], v[128:131]\n\t" \
  "v_mfma_f32_16x16x32_bf16 v[132:135], v[64:67], v[80:83], v[132:135]\n\t" \
  "ds_read_b64_tr_b16 v[48:49], %[wh1] offset:0\n\t" \
  "v_mfma_f32_16x16x32_bf16 v[136:139], v[68:71], v[80:83], v[136:139]\n\t" \
  "ds_read_b64_tr_b16 v[50:51], %[wh1] offset:512\n\t" \
  "v_mfma_f32_16x16x32_bf16 v[140:143], v[72:75], v[80:83], v[140:143]\n\t" \
  "ds_read_b128 v[80:83], %[xa0] offset:43008\n\t" \
  "v_mfma_f32_16x16x32_bf16 v[144:147], v[60:63], v[84:87], v[144:147]\n\t" \
  "v_mfma_f32_16x16x32_bf16 v[148:151], v[64:67], v[84:87], v[148:151]\n\t" \
  "ds_read_b64_tr_b16 v[52:53], %[wh2] offset:0\n\t" \
  "v_mfma_f32_16x16x32_bf16 v[152:155], v[68:71], v[84:87], v[152:155]\n\t" \
  "ds_read_b64_tr_b16 v[54:55], %[wh2] offset:512\n\t" \
  "v_mfma_f32_16x16x32_bf16 v[156:159], v[72:75], v[84:87], v[156:159]\n\t" \
  "ds_read_b128 v[84:87], %[xa0] offset:45056\n\t" \
  "v_mfma_f32_16x16x32_bf16 v[160:163], v[60:63], v[88:91], v[160:163]\n\t" \
  "v_mfma_f32_16x16x32_bf16 v[164:167], v[64:67], v[88:91], v[164:167]\n\t" \
  "ds_read_b64_tr_b16 v[56:57], %[wh3] offset:0\n\t" \
  "v_mfma_f32_16x16x32_bf16 v[168:171], v[68:71], v[88:91], v[168:171]\n\t" \
  "ds_read_b64_tr_b16 v[58:59], %[wh3] offset:512\n\t" \
  "v_mfma_f32_16x16x32_bf16 v[172:175], v[72:75], v[88:91], v[172:175]\n\t" \
  "ds_read_b128 v[88:91], %[xa0] offset:47104\n\t" \
  "v_mfma_f32_16x16x32_bf16 v[176:179], v[60:63], v[92:95], v[176:179]\n\t" \
  "v_mfma_f32_16x16x32_bf16 v[180:183], v[64:67], v[92:95], v[180:183]\n\t" \
  "ds_read_b128 v[104:107], %[xa0] offset:55296\n\t" \
  "v_mfma_f32_16x16x32_bf16 v[184:187], v[68:71], v[92:95], v[184:187]\n\t" \
  "v_mfma_f32_16x16x32_bf16 v[188:191], v[72:75], v[92:95], v[188:191]\n\t" \
  "ds_read_b128 v[92:95], %[xa0] offset:49152\n\t" \
  "v_mfma_f32_16x16x32_bf16 v[192:195], v[60:63], v[96:99], v[192:195]\n\t" \
  "v_mfma_f32_16x16x32_bf16 v[196:199], v[64:67], v[96:99], v[196:199]\n\t" \
  "ds_read_b128 v[108:111], %[xa0] offset:57344\n\t" \
  "v_mfma_f32_16x16x32_bf16 v[200:203], v[68:71], v[96:99], v[200:203]\n\t" \
  "v_mfma_f32_16x16x32_bf16 v[204:207], v[72:75], v[96:99], v[204:207]\n\t" \
  "ds_read_b128 v[96:99], %[xa0] offset:51200\n\t" \
  "v_mfma_f32_16x16x32_bf16 v[208:211], v[60:63], v[100:103], v[208:211]\n\t" \
  "v_mfma_f32_16x16x32_bf16 v[212:215], v[64:67], v[100:103], v[212:215]\n\t" \
  "ds_read_b128 v[32:35], %[xa0] offset:59392\n\t" \
  "v_mfma_f32_16x16x32_bf16 v[216:219], v[68:71], v[100:103], v[216:219]\n\t" \
  "v_mfma_f32_16x16x32_bf16 v[220:223], v[72:75], v[100:103], v[220:223]\n\t" \
  "ds_read_b128 v[100:103], %[xa0] offset:53248\n\t" \
  "v_mfma_f32_16x16x32_bf16 v[224:227], v[60:63], v[36:39], v[224:227]\n\t" \
  "v_mfma_f32_16x16x32_bf16 v[228:231], v[64:67], v[36:39], v[228:231]\n\t" \
  "v_mfma_f32_16x16x32_bf16 v[232:235], v[68:71], v[36:39], v[232:235]\n\t" \
  "v_mfma_f32_16x16x32_bf16 v[236:239], v[72:75], v[36:39], v[236:239]\n\t" \
  "v_mfma_f32_16x16x32_bf16 v[240:243], v[60:63], v[40:43], v[240:243]\n\t" \
  "v_mfma_f32_16x16x32_bf16 v[244:247], v[64:67], v[40:43], v[244:247]\n\t" \
  "v_mfma_f32_16x16x32_bf16 v[248:251], v[68:71], v[40:43], v[248:251]\n\t" \
  "v_mfma_f32_16x16x32_bf16 v[252:255], v[72:75], v[40:43], v[252:255]\n\t" \
  "s_waitcnt lgkmcnt(0)\n\t" \
  "v_mfma_f32_16x16x32_bf16 v[28:31], v[44:47], v[32:35], v[28:31]\n\t" \
  "v_mfma_f32_16x16x32_bf16 v[112:115], v[44:47], v[76:79], v[112:115]\n\t" \
  "v_mfma_f32_16x16x32_bf16 v[116:119], v[48:51], v[76:79], v[116:119]\n\t" \
  "ds_read_b64_tr_b16 v[60:61], %[wh0] offset:16384\n\t" \
  "v_mfma_f32_16x16x32_bf16 v[120:123], v[52:55], v[76:79], v[120:123]\n\t" \
  "ds_read_b64_tr_b16 v[62:63], %[wh0] offset:16896\n\t" \
  "v_mfma_f32_16x16x32_bf16 v[124:127], v[56:59], v[76:79], v[124:127]\n\t" \
  "ds_read_b128 v[76:79], %[xa1] offset:40960\n\t" \
  "v_mfma_f32_16x16x32_bf16 v[128:131], v[44:47], v[80:83], v[128:131]\n\t" \
  "v_mfma_f32_16x16x32_bf16 v[132:135], v[48:51], v[80:83], v[132:135]\n\t" \
  "ds_read_b64_tr_b16 v[64:65], %[wh1] offset:16384\n\t" \
  "v_mfma_f32_16x16x32_bf16 v[136:139], v[52:55], v[80:83], v[136:139]\n\t" \
  "ds_read_b64_tr_b16 v[66:67], %[wh1] offset:16896\n\t" \
  "v_mfma_f32_16x16x32_bf16 v[140:143], v[56:59], v[80:83], v[140:143]\n\t" \
  "ds_read_b128 v[80:83], %[xa1] offset:43008\n\t" \
  "v_mfma_f32_16x16x32_bf16 v[144:147], v[44:47], v[84:87], v[144:147]\n\t" \
  "v_mfma_f32_16x16x32_bf16 v[148:151], v[48:51], v[84:87], v[148:151]\n\t" \
  "ds_read_b64_tr_b16 v[68:69], %[wh2] offset:16384\n\t" \
  "v_mfma_f32_16x16x32_bf16 v[152:155], v[52:55], v[84:87], v[152:155]\n\t" \
  "ds_read_b64_tr_b16 v[70:71], %[wh2] offset:16896\n\t" \
  "v_mfma_f32_16x16x32_bf16 v[156:159], v[56:59], v[84:87], v[156:159]\n\t" \
  "ds_read_b128 v[84:87], %[xa1] offset:45056\n\t" \
  "v_mfma_f32_16x16x32_bf16 v[160:163], v[44:47], v[88:91], v[160:163]\n\t" \
  "v_mfma_f32_16x16x32_bf16 v[164:167], v[48:51], v[88:91], v[164:167]\n\t" \
  "ds_read_b64_tr_b16 v[72:73], %[wh3] offset:16384\n\t" \
  "v_mfma_f32_16x16x32_bf16 v[168:171], v[52:55], v[88:91], v[168:171]\n\t" \
  "ds_read_b64_tr_b16 v[74:75], %[wh3] offset:16896\n\t" \
  "v_mfma_f32_16x16x32_bf16 v[172:175], v[56:59], v[88:91], v[172:175]\n\t" \
  "ds_read_b128 v[88:91], %[xa1] offset:47104\n\t" \
  "v_mfma_f32_16x16x32_bf16 v[176:179], v[44:47], v[92:95], v[176:179]\n\t" \
  "v_mfma_f32_16x16x32_bf16 v[180:183], v[48:51], v[92:95], v[180:183]\n\t" \
  "ds_read_b128 v[36:39], %[xa1] offset:55296\n\t" \
  "v_mfma_f32_16x16x32_bf16 v[184:187], v[52:55], v[92:95], v[184:187]\n\t" \
  "v_mfma_f32_16x16x32_bf16 v[188:191], v[56:59], v[92:95], v[188:191]\n\t" \
  "ds_read_b128 v[92:95], %[xa1] offset:49152\n\t" \
  "v_mfma_f32_16x16x32_bf16 v[192:195], v[44:47], v[96:99], v[192:195]\n\t" \
  "v_mfma_f32_16x16x32_bf16 v[196:199], v[48:51], v[96:99], v[196:199]\n\t" \
  "ds_read_b128 v[40:43], %[xa1] offset:57344\n\t" \
  "v_mfma_f32_16x16x32_bf16 v[200:203], v[52:55], v[96:99], v[200:203]\n\t" \
  "v_mfma_f32_16x16x32_bf16 v[204:207], v[56:59], v[96:99], v[204:207]\n\t" \
  "ds_read_b128 v[96:99], %[xa1] offset:51200\n\t" \
  "v_mfma_f32_16x16x32_bf16 v[208:211], v[44:47], v[100:103], v[208:211]\n\t" \
  "v_mfma_f32_16x16x32_bf16 v[212:215], v[48:51], v[100:103], v[212:215]\n\t" \
  "ds_read_b128 v[32:35], %[xa1] offset:59392\n\t" \
  "v_mfma_f32_16x16x32_bf16 v[216:219], v[52:55], v[100:103], v[216:219]\n\t" \
  "v_mfma_f32_16x16x32_bf16 v[220:223], v[56:59], v[100:103], v[220:223]\n\t" \
  "ds_read_b128 v[100:103], %[xa1] offset:53248\n\t" \
  "v_mfma_f32_16x16x32_bf16 v[224:227], v[44:47], v[104:107], v[224:227]\n\t" \
  "v_mfma_f32_16x16x32_bf16 v[228:231], v[48:51], v[104:107], v[228:231]\n\t" \
  "v_mfma_f32_16x16x32_bf16 v[232:235], v[52:55], v[104:107], v[232:235]\n\t" \
  "v_mfma_f32_16x16x32_bf16 v[236:239], v[56:59], v[104:107], v[236:239]\n\t" \
  "v_mfma_f32_16x16x32_bf16 v[240:243], v[44:47], v[108:111], v[240:243]\n\t" \
  "v_mfma_f32_16x16x32_bf16 v[244:247], v[48:51], v[108:111], v[244:247]\n\t" \
  "v_mfma_f32_16x16x32_bf16 v[248:251], v[52:55], v[108:111], v[248:251]\n\t" \
  "v_mfma_f32_16x16x32_bf16 v[252:255], v[56:59], v[108:111], v[252:255]\n\t" \
  "s_waitcnt lgkmcnt(0)\n\t" \
  "v_mfma_f32_16x16x32_bf16 v[28:31], v[60:63], v[32:35], v[28:31]\n\t" \
  "s_barrier\n\t" \
  "v_mfma_f32_16x16x32_bf16 v[112:115], v[60:63], v[76:79], v[112:115]\n\t" \
  "v_mfma_f32_16x16x32_bf16 v[116:119], v[64:67], v[76:79], v[116:119]\n\t" \
  "ds_read_b64_tr_b16 v[44:45], %[wl0] offset:0\n\t" \
  "v_mfma_f32_16x16x32_bf16 v[120:123], v[68:71], v[76:79], v[120:123]\n\t" \
  "ds_read_b64_tr_b16 v[46:47], %[wl0] offset:512\n\t" \
  "v_mfma_f32_16x16x32_bf16 v[124:127], v[72:75], v[76:79], v[124:127]\n\t" \
  "ds_read_b128 v[76:79], %[xa0] offset:0\n\t" \
  "v_mfma_f32_16x16x32_bf16 v[128:131], v[60:63], v[80:83], v[128:131]\n\t" \
  "v_mfma_f32_16x16x32_bf16 v[132:135], v[64:67], v[80:83], v[132:135]\n\t" \
  "ds_read_b64_tr_b16 v[48:49], %[wl1] offset:0\n\t" \
  "v_mfma_f32_16x16x32_bf16 v[136:139], v[68:71], v[80:83], v[136:139]\n\t" \
  "ds_read_b64_tr_b16 v[50:51], %[wl1] offset:512\n\t" \
  "v_mfma_f32_16x16x32_bf16 v[140:143], v[72:75], v[80:83], v[140:143]\n\t" \
  "ds_read_b128 v[80:83], %[xa0] offset:2048\n\t" \
  "v_mfma_f32_16x16x32_bf16 v[144:147], v[60:63], v[84:87], v[144:147]\n\t" \
  "v_mfma_f32_16x16x32_bf16 v[148:151], v[64:67], v[84:87], v[148:151]\n\t" \
  "ds_read_b64_tr_b16 v[52:53], %[wl2] offset:0\n\t" \
  "v_mfma_f32_16x16x32_bf16 v[152:155], v[68:71], v[84:87], v[152:155]\n\t" \
  "ds_read_b64_tr_b16 v[54:55], %[wl2] offset:512\n\t" \
  "v_mfma_f32_16x16x32_bf16 v[156:159], v[72:75], v[84:87], v[156:159]\n\t" \
  "ds_read_b128 v[84:87], %[xa0] offset:4096\n\t" \
  "v_mfma_f32_16x16x32_bf16 v[160:163], v[60:63], v[88:91], v[160:163]\n\t" \
  "v_mfma_f32_16x16x32_bf16 v[164:167], v[64:67], v[88:91], v[164:167]\n\t" \
  "ds_read_b64_tr_b16 v[56:57], %[wl3] offset:0\n\t" \
  "v_mfma_f32_16x16x32_bf16 v[168:171], v[68:71], v[88:91], v[168:171]\n\t" \
  "ds_read_b64_tr_b16 v[58:59], %[wl3] offset:512\n\t" \
  "v_mfma_f32_16x16x32_bf16 v[172:175], v[72:75], v[88:91], v[172:175]\n\t" \
  "ds_read_b128 v[88:91], %[xa0] offset:6144\n\t" \
  "v_mfma_f32_16x16x32_bf16 v[176:179], v[60:63], v[92:95], v[176:179]\n\t" \
  "v_mfma_f32_16x16x32_bf16 v[180:183], v[64:67], v[92:95], v[180:183]\n\t" \
  "ds_read_b128 v[104:107], %[xa0] offset:14336\n\t" \
  "v_mfma_f32_16x16x32_bf16 v[184:187], v[68:71], v[92:95], v[184:187]\n\t" \
  "v_mfma_f32_16x16x32_bf16 v[188:191], v[72:75], v[92:95], v[188:191]\n\t" \
  "ds_read_b128 v[92:95], %[xa0] offset:8192\n\t" \
  "v_mfma_f32_16x16x32_bf16 v[192:195], v[60:63], v[96:99], v[192:195]\n\t" \
  "v_mfma_f32_16x16x32_bf16 v[196:199], v[64:67], v[96:99], v[196:199]\n\t" \
  "ds_read_b128 v[108:111], %[xa0] offset:16384\n\t" \
  "v_mfma_f32_16x16x32_bf16 v[200:203], v[68:71], v[96:99], v[200:203]\n\t" \
  "v_mfma_f32_16x16x32_bf16 v[204:207], v[72:75], v[96:99], v[204:207]\n\t" \
  "ds_read_b128 v[96:99], %[xa0] offset:10240\n\t" \
  "v_mfma_f32_16x16x32_bf16 v[208:211], v[60:63], v[100:103], v[208:211]\n\t" \
  "v_mfma_f32_16x16x32_bf16 v[212:215], v[64:67], v[100:103], v[212:215]\n\t" \
  "ds_read_b128 v[32:35], %[xa0] offset:18432\n\t" \
  "v_mfma_f32_16x16x32_bf16 v[216:219], v[68:71], v[100:103], v[216:219]\n\t" \
  "v_mfma_f32_16x16x32_bf16 v[220:223], v[72:75], v[100:103], v[220:223]\n\t" \
  "ds_read_b128 v[100:103], %[xa0] offset:12288\n\t" \
  "v_mfma_f32_16x16x32_bf16 v[224:227], v[60:63], v[36:39], v[224:227]\n\t" \
  "v_mfma_f32_16x16x32_bf16 v[228:231], v[64:67], v[36:39], v[228:231]\n\t" \
  "v_mfma_f32_16x16x32_bf16 v[232:235], v[68:71], v[36:39], v[232:235]\n\t" \
  "s_sub_u32 %[n], %[n], 1\n\t" \
  "v_mfma_f32_16x16x32_bf16 v[236:239], v[72:75], v[36:39], v[236:239]\n\t" \
  "s_cmp_lg_u32 %[n], 0\n\t" \
  "v_mfma_f32_16x16x32_bf16 v[240:243], v[60:63], v[40:43], v[240:243]\n\t" \
  "v_mfma_f32_16x16x32_bf16 v[244:247], v[64:67], v[40:43], v[244:247]\n\t" \
  "v_mfma_f32_16x16x32_bf16 v[248:251], v[68:71], v[40:43], v[248:251]\n\t" \
  "v_mfma_f32_16x16x32_bf16 v[252:255], v[72:75], v[40:43], v[252:255]\n\t" \
  "s_cbranch_scc1 Lloop_%=\n\t" \
  "s_waitcnt lgkmcnt(0)\n\t" \
  "v_mfma_f32_16x16x32_bf16 v[28:31], v[44:47], v[32:35], v[28:31]\n\t" \
  "v_mfma_f32_16x16x32_bf16 v[112:115], v[44:47], v[76:79], v[112:115]\n\t" \
  "v_mfma_f32_16x16x32_bf16 v[116:119], v[48:51], v[76:79], v[116:119]\n\t" \
  "ds_read_b64_tr_b16 v[60:61], %[wl0] offset:16384\n\t" \
  "v_mfma_f32_16x16x32_bf16 v[120:123], v[52:55], v[76:79], v[120:123]\n\t" \
  "ds_read_b64_tr_b16 v[62:63], %[wl0] offset:16896\n\t" \
  "v_mfma_f32_16x16x32_bf16 v[124:127], v[56:59], v[76:79], v[124:127]\n\t" \
  "ds_read_b128 v[76:79], %[xa1] offset:0\n\t" \
  "v_mfma_f32_16x16x32_bf16 v[128:131], v[44:47], v[80:83], v[128:131]\n\t" \
  "v_mfma_f32_16x16x32_bf16 v[132:135], v[48:51], v[80:83], v[132:135]\n\t" \
  "ds_read_b64_tr_b16 v[64:65], %[wl1] offset:16384\n\t" \
  "v_mfma_f32_16x16x32_bf16 v[136:139], v[52:55], v[80:83], v[136:139]\n\t" \
  "ds_read_b64_tr_b16 v[66:67], %[wl1] offset:16896\n\t" \
  "v_mfma_f32_16x16x32_bf16 v[140:143], v[56:59], v[80:83], v[140:143]\n\t" \
  "ds_read_b128 v[80:83], %[xa1] offset:2048\n\t" \
  "v_mfma_f32_16x16x32_bf16 v[144:147], v[44:47], v[84:87], v[144:147]\n\t" \
  "v_mfma_f32_16x16x32_bf16 v[148:151], v[48:51], v[84:87], v[148:151]\n\t" \
  "ds_read_b64_tr_b16 v[68:69], %[wl2] offset:16384\n\t" \
  "v_mfma_f32_16x16x32_bf16 v[152:155], v[52:55], v[84:87], v[152:155]\n\t" \
  "ds_read_b64_tr_b16 v[70:71], %[wl2] offset:16896\n\t" \
  "v_mfma_f32_16x16x32_bf16 v[156:159], v[56:59], v[84:87], v[156:159]\n\t" \
  "ds_read_b128 v[84:87], %[xa1] offset:4096\n\t" \
  "v_mfma_f32_16x16x32_bf16 v[160:163], v[44:47], v[88:91], v[160:163]\n\t" \
  "v_mfma_f32_16x16x32_bf16 v[164:167], v[48:51], v[88:91], v[164:167]\n\t" \
  "ds_read_b64_tr_b16 v[72:73], %[wl3] offset:16384\n\t" \
  "v_mfma_f32_16x16x32_bf16 v[168:171], v[52:55], v[88:91], v[168:171]\n\t" \
  "ds_read_b64_tr_b16 v[74:75], %[wl3] offset:16896\n\t" \
  "v_mfma_f32_16x16x32_bf16 v[172:175], v[56:59], v[88:91], v[172:175]\n\t" \
  "ds_read_b128 v[88:91], %[xa1] offset:6144\n\t" \
  "v_mfma_f32_16x16x32_bf16 v[176:179], v[44:47], v[92:95], v[176:179]\n\t" \
  "v_mfma_f32_16x16x32_bf16 v[180:183], v[48:51], v[92:95], v[180:183]\n\t" \
  "ds_read_b128 v[36:39], %[xa1] offset:14336\n\t" \
  "v_mfma_f32_16x16x32_bf16 v[184:187], v[52:55], v[92:95], v[184:187]\n\t" \
  "v_mfma_f32_16x16x32_bf16 v[188:191], v[56:59], v[92:95], v[188:191]\n\t" \
  "ds_read_b128 v[92:95], %[xa1] offset:8192\n\t" \
  "v_mfma_f32_16x16x32_bf16 v[192:195], v[44:47], v[96:99], v[192:195]\n\t" \
  "v_mfma_f32_16x16x32_bf16 v[196:199], v[48:51], v[96:99], v[196:199]\n\t" \
  "ds_read_b128 v[40:43], %[xa1] offset:16384\n\t" \
  "v_mfma_f32_16x16x32_bf16 v[200:203], v[52:55], v[96:99], v[200:203]\n\t" \
  "v_mfma_f32_16x16x32_bf16 v[204:207], v[56:59], v[96:99], v[204:207]\n\t" \
  "ds_read_b128 v[96:99], %[xa1] offset:10240\n\t" \
  "v_mfma_f32_16x16x32_bf16 v[208:211], v[44:47], v[100:103], v[208:211]\n\t" \
  "v_mfma_f32_16x16x32_bf16 v[212:215], v[48:51], v[100:103], v[212:215]\n\t" \
  "ds_read_b128 v[32:35], %[xa1] offset:18432\n\t" \
  "v_mfma_f32_16x16x32_bf16 v[216:219], v[52:55], v[100:103], v[216:219]\n\t" \
  "v_mfma_f32_16x16x32_bf16 v[220:223], v[56:59], v[100:103], v[220:223]\n\t" \
  "ds_read_b128 v[100:103], %[xa1] offset:12288\n\t" \
  "ds_read_b32 %[bd], %[bd]\n\t" \
  "v_mfma_f32_16x16x32_bf16 v[224:227], v[44:47], v[104:107], v[224:227]\n\t" \
  "v_mfma_f32_16x16x32_bf16 v[228:231], v[48:51], v[104:107], v[228:231]\n\t" \
  "v_mfma_f32_16x16x32_bf16 v[232:235], v[52:55], v[104:107], v[232:235]\n\t" \
  "v_mfma_f32_16x16x32_bf16 v[236:239], v[56:59], v[104:107], v[236:239]\n\t" \
  "v_mfma_f32_16x16x32_bf16 v[240:243], v[44:47], v[108:111], v[240:243]\n\t" \
  "v_mfma_f32_16x16x32_bf16 v[244:247], v[48:51], v[108:111], v[244:247]\n\t" \
  "v_mfma_f32_16x16x32_bf16 v[248:251], v[52:55], v[108:111], v[248:251]\n\t" \
  "v_mfma_f32_16x16x32_bf16 v[252:255], v[56:59], v[108:111], v[252:255]\n\t" \
  "s_waitcnt lgkmcnt(0)\n\t" \
  "s_waitcnt vmcnt(0)\n\t" \
  "s_nop 7\n\t" \
  "s_nop 7\n\t" \
  "s_nop 7\n\t" \
  ""
#define CONS_CLOB "v44", "v45", "v46", "v47", "v48", "v49", "v50", "v51", "v52", "v53", "v54", "v55", "v56", "v57", "v58", "v59", "v108", "v109", "v110", "v111"
#define FAST_ASM \
  "Lf0_%=:\n\t" \
  "s_waitcnt vmcnt(7)\n\t" \
  "global_load_dwordx3 v[228:230], %[o12], %[pk] offset:2048 sc0 sc1\n\t" \
  "global_load_dwordx4 v[220:223], %[o1], %[pk] sc0 sc1\n\t" \
  "global_load_dwordx4 v[224:227], %[o1], %[pk] offset:1024 sc0 sc1\n\t" \
  "v_lshrrev_b32 v64, 1, v244\n\t" \
  "v_lshrrev_b32 v65, 1, v245\n\t" \
  "v_bitop3_b32 v64, v244, %[k55], v64 bitop3:0x80\n\t" \
  "v_bitop3_b32 v65, v245, %[k55], v65 bitop3:0x80\n\t" \
  "v_bcnt_u32_b32 v66, v64, 0\n\t" \
  "v_bcnt_u32_b32 v67, v65, v66\n\t" \
  "v_and_b32 v70, 0x7070707, v246\n\t" \
  "v_lshrrev_b32 v71, 4, v246\n\t" \
  "v_add_u32_dpp v68, v67, v67 row_shr:1 row_mask:0xf bank_mask:0xf bound_ctrl:1\n\t" \
  "v_perm_b32 v70, %[l1], %[l0], v70\n\t" \
  "v_and_b32 v71, 0x7070707, v71\n\t" \
  "v_add_u32_dpp v68, v68, v68 row_shr:2 row_mask:0xf bank_mask:0xf bound_ctrl:1\n\t" \
  "ds_write_b16 %[wa], v70\n\t" \
  "ds_write_b16_d16_hi %[wa], v70 offset:2\n\t" \
  "v_add_u32_dpp v68, v68, v68 row_shr:4 row_mask:0xf bank_mask:0xf bound_ctrl:1\n\t" \
  "v_perm_b32 v71, %[l1], %[l0], v71\n\t" \
  "v_lshrrev_b32 v69, 3, v246\n\t" \
  "v_add_u32_dpp v68, v68, v68 row_shr:8 row_mask:0xf bank_mask:0xf bound_ctrl:1\n\t" \
  "v_and_b32 v69, 0x1110111, v69\n\t" \
  "ds_write_b16 %[wa], v71 offset:4\n\t" \
  "v_add_u32_dpp v68, v68, v68 row_bcast:15 row_mask:0xa bank_mask:0xf\n\t" \
  "v_pk_mul_lo_u16 v69, v69, %[k124] op_sel_hi:[1,0]\n\t" \
  "ds_write_b16_d16_hi %[wa], v71 offset:6\n\t" \
  "v_add_u32_dpp v68, v68, v68 row_bcast:31 row_mask:0xc bank_mask:0xf\n\t" \
  "v_perm_b32 v69, v69, v69, %[kc31]\n\t" \
  "v_sub_u32 v67, %[scr], v67\n\t" \
  "v_and_b32 v69, 0x707, v69\n\t" \
  "s_lshr_b32 %[s0], %[h0], 8\n\t" \
  "v_perm_b32 v69, %[l1], %[l0], v69\n\t" \
  "s_lshr_b32 %[s1], %[h0], 16\n\t" \
  "ds_write_b16 %[wa], v69 offset:8\n\t" \
  "v_and_or_b32 v70, v246, %[k7ff], %[scr]\n\t" \
  "s_bfm_b64 exec, %[s1], %[s0]\n\t" \
  "ds_write_b8_d16_hi v70, v246\n\t" \
  "s_mov_b64 exec, -1\n\t" \
  "v_add_u32 v72, v68, v67\n\t" \
  "v_add_u32 v76, v72, v66\n\t" \
  "v_and_b32 v80, -4, v72\n\t" \
  "ds_read2_b32 v[88:89], v80 offset1:1\n\t" \
  "v_and_b32 v73, 0x1010101, v64\n\t" \
  "v_bcnt_u32_b32 v73, v73, v72\n\t" \
  "v_and_b32 v81, -4, v73\n\t" \
  "ds_read2_b32 v[90:91], v81 offset1:1\n\t" \
  "v_and_b32 v74, 0x5050505, v64\n\t" \
  "v_bcnt_u32_b32 v74, v74, v72\n\t" \
  "v_and_b32 v82, -4, v74\n\t" \
  "ds_read2_b32 v[92:93], v82 offset1:1\n\t" \
  "v_and_b32 v75, 0x15151515, v64\n\t" \
  "v_bcnt_u32_b32 v75, v75, v72\n\t" \
  "v_and_b32 v83, -4, v75\n\t" \
  "ds_read2_b32 v[94:95], v83 offset1:1\n\t" \
  "v_and_b32 v84, -4, v76\n\t" \
  "ds_read2_b32 v[96:97], v84 offset1:1\n\t" \
  "v_and_b32 v77, 0x1010101, v65\n\t" \
  "v_bcnt_u32_b32 v77, v77, v76\n\t" \
  "v_and_b32 v85, -4, v77\n\t" \
  "ds_read2_b32 v[98:99], v85 offset1:1\n\t" \
  "v_and_b32 v78, 0x5050505, v65\n\t" \
  "v_bcnt_u32_b32 v78, v78, v76\n\t" \
  "v_and_b32 v86, -4, v78\n\t" \
  "ds_read2_b32 v[100:101], v86 offset1:1\n\t" \
  "v_and_b32 v79, 0x15151515, v65\n\t" \
  "v_bcnt_u32_b32 v79, v79, v76\n\t" \
  "v_and_b32 v87, -4, v79\n\t" \
  "ds_read2_b32 v[102:103], v87 offset1:1\n\t" \
  "s_lshr_b32 %[s0], %[up], 6\n\t" \
  "v_lshl_or_b32 v120, %[s0], %[fsh], %[vf]\n\t" \
  "s_waitcnt vmcnt(5)\n\t" \
  "s_lshl_b64 exec, 1, %[up]\n\t" \
  "global_store_dword v120, %[vcall], %[rdy] sc0 sc1\n\t" \
  "s_mov_b64 exec, -1\n\t" \
  "v_lshlrev_b32 v70, 8, v64\n\t" \
  "v_lshl_add_u32 v69, v64, 1, v64\n\t" \
  "v_lshl_add_u32 v71, v70, 8, v70\n\t" \
  "v_lshl_add_u32 v71, v70, 16, v71\n\t" \
  "v_bfi_b32 v71, v69, v71, v244\n\t" \
  "v_lshlrev_b32 v70, 2, v64\n\t" \
  "v_bfi_b32 v70, %[k33], v71, v70\n\t" \
  "v_lshrrev_b32 v71, 2, v71\n\t" \
  "v_bfi_b32 v71, %[k33], v71, v64\n\t" \
  "v_and_b32 v80, 0x7070707, v70\n\t" \
  "v_and_b32 v81, 0x7070707, v71\n\t" \
  "v_lshrrev_b32 v70, 4, v70\n\t" \
  "v_lshrrev_b32 v71, 4, v71\n\t" \
  "v_and_b32 v82, 0x7070707, v70\n\t" \
  "v_and_b32 v83, 0x7070707, v71\n\t" \
  "v_lshlrev_b32 v70, 8, v65\n\t" \
  "v_lshl_add_u32 v69, v65, 1, v65\n\t" \
  "v_lshl_add_u32 v71, v70, 8, v70\n\t" \
  "v_lshl_add_u32 v71, v70, 16, v71\n\t" \
  "v_bfi_b32 v71, v69, v71, v245\n\t" \
  "v_lshlrev_b32 v70, 2, v65\n\t" \
  "v_bfi_b32 v70, %[k33], v71, v70\n\t" \
  "v_lshrrev_b32 v71, 2, v71\n\t" \
  "v_bfi_b32 v71, %[k33], v71, v65\n\t" \
  "v_and_b32 v84, 0x7070707, v70\n\t" \
  "v_and_b32 v85, 0x7070707, v71\n\t" \
  "v_lshrrev_b32 v70, 4, v70\n\t" \
  "v_lshrrev_b32 v71, 4, v71\n\t" \
  "v_and_b32 v86, 0x7070707, v70\n\t" \
  "v_and_b32 v87, 0x7070707, v71\n\t" \
  "s_waitcnt lgkmcnt(0)\n\t" \
  "v_alignbyte_b32 v69, v89, v88, v72\n\t" \
  "v_perm_b32 v69, v69, %[lp], v80\n\t" \
  "v_bfi_b32 v70, %[k80], v69, v236\n\t" \
  "v_bfi_b32 v71, %[k80], v236, v69\n\t" \
  "v_perm_b32 v104, v71, v70, %[kp0]\n\t" \
  "v_perm_b32 v105, v71, v70, %[kp1]\n\t" \
  "v_alignbyte_b32 v69, v91, v90, v73\n\t" \
  "v_perm_b32 v69, v69, %[lp], v81\n\t" \
  "v_bfi_b32 v70, %[k80], v69, v237\n\t" \
  "v_bfi_b32 v71, %[k80], v237, v69\n\t" \
  "v_perm_b32 v106, v71, v70, %[kp0]\n\t" \
  "v_perm_b32 v107, v71, v70, %[kp1]\n\t" \
  "global_store_dwordx4 %[o0], v[104:107], %[xb] offset:0 sc0 sc1\n\t" \
  "v_alignbyte_b32 v69, v93, v92, v74\n\t" \
  "v_perm_b32 v69, v69, %[lp], v82\n\t" \
  "v_bfi_b32 v70, %[k80], v69, v238\n\t" \
  "v_bfi_b32 v71, %[k80], v238, v69\n\t" \
  "v_perm_b32 v108, v71, v70, %[kp0]\n\t" \
  "v_perm_b32 v109, v71, v70, %[kp1]\n\t" \
  "v_alignbyte_b32 v69, v95, v94, v75\n\t" \
  "v_perm_b32 v69, v69, %[lp], v83\n\t" \
  "v_bfi_b32 v70, %[k80], v69, v239\n\t" \
  "v_bfi_b32 v71, %[k80], v239, v69\n\t" \
  "v_perm_b32 v110, v71, v70, %[kp0]\n\t" \
  "v_perm_b32 v111, v71, v70, %[kp1]\n\t" \
  "global_store_dwordx4 %[o0], v[108:111], %[xb] offset:1024 sc0 sc1\n\t" \
  "v_alignbyte_b32 v69, v97, v96, v76\n\t" \
  "v_perm_b32 v69, v69, %[lp], v84\n\t" \
  "v_bfi_b32 v70, %[k80], v69, v240\n\t" \
  "v_bfi_b32 v71, %[k80], v240, v69\n\t" \
  "v_perm_b32 v112, v71, v70, %[kp0]\n\t" \
  "v_perm_b32 v113, v71, v70, %[kp1]\n\t" \
  "v_alignbyte_b32 v69, v99, v98, v77\n\t" \
  "v_perm_b32 v69, v69, %[lp], v85\n\t" \
  "v_bfi_b32 v70, %[k80], v69, v241\n\t" \
  "v_bfi_b32 v71, %[k80], v241, v69\n\t" \
  "v_perm_b32 v114, v71, v70, %[kp0]\n\t" \
  "v_perm_b32 v115, v71, v70, %[kp1]\n\t" \
  "global_store_dwordx4 %[o0], v[112:115], %[xb] offset:2048 sc0 sc1\n\t" \
  "v_alignbyte_b32 v69, v101, v100, v78\n\t" \
  "v_perm_b32 v69, v69, %[lp], v86\n\t" \
  "v_bfi_b32 v70, %[k80], v69, v242\n\t" \
  "v_bfi_b32 v71, %[k80], v242, v69\n\t" \
  "v_perm_b32 v116, v71, v70, %[kp0]\n\t" \
  "v_perm_b32 v117, v71, v70, %[kp1]\n\t" \
  "v_alignbyte_b32 v69, v103, v102, v79\n\t" \
  "v_perm_b32 v69, v69, %[lp], v87\n\t" \
  "v_bfi_b32 v70, %[k80], v69, v243\n\t" \
  "v_bfi_b32 v71, %[k80], v243, v69\n\t" \
  "v_perm_b32 v118, v71, v70, %[kp0]\n\t" \
  "v_perm_b32 v119, v71, v70, %[kp1]\n\t" \
  "global_store_dwordx4 %[o0], v[116:119], %[xb] offset:3072 sc0 sc1\n\t" \
  "s_waitcnt vmcnt(8)\n\t" \
  "s_mov_b32 %[up], %[u0]\n\t" \
  "v_readfirstlane_b32 %[u0], v254\n\t" \
  "s_min_u32 %[u0], %[u0], " TK_MAX "\n\t" \
  "" TK_ADD " %[u0], %[u0], %[nws]\n\t" \
  "s_sub_u32 %[s0], %[u0], %[wb]\n\t" \
  "v_readlane_b32 %[h0], v252, %[s0]\n\t" \
  "v_readlane_b32 %[hc], v253, %[s0]\n\t" \
  "s_add_u32 %[wb], %[u0], 1\n\t" \
  "v_lshl_add_u32 v71, %[wb], 3, %[v8]\n\t" \
  "s_or_b32 %[s0], %[u0], %[tag]\n\t" \
  "global_load_dwordx2 v[252:253], v71, %[hdr] sc0 sc1\n\t" \
  "s_mov_b64 exec, 1\n\t" \
  "global_atomic_add v254, %[v16], %[one], %[tkt] sc0 sc1\n\t" \
  "s_mov_b64 exec, -1\n\t" \
  "v_lshl_add_u32 %[o0], %[u0], 12, %[v16]\n\t" \
  "v_lshl_add_u32 %[o12], %[u0], 12, %[v12]\n\t" \
  "s_and_b32 %[s1], %[h0], 0x830000ff\n\t" \
  "s_cmp_eq_u32 %[hc], %[s0]\n\t" \
  "s_cselect_b32 %[s1], %[s1], 0\n\t" \
  "s_cmp_eq_u32 %[s1], %[pfk]\n\t" \
  "s_cbranch_scc0 Lf2_%=\n\t" \
  "s_waitcnt vmcnt(7)\n\t" \
  "global_load_dwordx3 v[244:246], %[o12], %[pk] offset:2048 sc0 sc1\n\t" \
  "global_load_dwordx4 v[236:239], %[o0], %[pk] sc0 sc1\n\t" \
  "global_load_dwordx4 v[240:243], %[o0], %[pk] offset:1024 sc0 sc1\n\t" \
  "v_lshrrev_b32 v64, 1, v228\n\t" \
  "v_lshrrev_b32 v65, 1, v229\n\t" \
  "v_bitop3_b32 v64, v228, %[k55], v64 bitop3:0x80\n\t" \
  "v_bitop3_b32 v65, v229, %[k55], v65 bitop3:0x80\n\t" \
  "v_bcnt_u32_b32 v66, v64, 0\n\t" \
  "v_bcnt_u32_b32 v67, v65, v66\n\t" \
  "v_and_b32 v70, 0x7070707, v230\n\t" \
  "v_lshrrev_b32 v71, 4, v230\n\t" \
  "v_add_u32_dpp v68, v67, v67 row_shr:1 row_mask:0xf bank_mask:0xf bound_ctrl:1\n\t" \
  "v_perm_b32 v70, %[l1], %[l0], v70\n\t" \
  "v_and_b32 v71, 0x7070707, v71\n\t" \
  "v_add_u32_dpp v68, v68, v68 row_shr:2 row_mask:0xf bank_mask:0xf bound_ctrl:1\n\t" \
  "ds_write_b16 %[wa], v70\n\t" \
  "ds_write_b16_d16_hi %[wa], v70 offset:2\n\t" \
  "v_add_u32_dpp v68, v68, v68 row_shr:4 row_mask:0xf bank_mask:0xf bound_ctrl:1\n\t" \
  "v_perm_b32 v71, %[l1], %[l0], v71\n\t" \
  "v_lshrrev_b32 v69, 3, v230\n\t" \
  "v_add_u32_dpp v68, v68, v68 row_shr:8 row_mask:0xf bank_mask:0xf bound_ctrl:1\n\t" \
  "v_and_b32 v69, 0x1110111, v69\n\t" \
  "ds_write_b16 %[wa], v71 offset:4\n\t" \
  "v_add_u32_dpp v68, v68, v68 row_bcast:15 row_mask:0xa bank_mask:0xf\n\t" \
  "v_pk_mul_lo_u16 v69, v69, %[k124] op_sel_hi:[1,0]\n\t" \
  "ds_write_b16_d16_hi %[wa], v71 offset:6\n\t" \
  "v_add_u32_dpp v68, v68, v68 row_bcast:31 row_mask:0xc bank_mask:0xf\n\t" \
  "v_perm_b32 v69, v69, v69, %[kc31]\n\t" \
  "v_sub_u32 v67, %[scr], v67\n\t" \
  "v_and_b32 v69, 0x707, v69\n\t" \
  "s_lshr_b32 %[s0], %[h1], 8\n\t" \
  "v_perm_b32 v69, %[l1], %[l0], v69\n\t" \
  "s_lshr_b32 %[s1], %[h1], 16\n\t" \
  "ds_write_b16 %[wa], v69 offset:8\n\t" \
  "v_and_or_b32 v70, v230, %[k7ff], %[scr]\n\t" \
  "s_bfm_b64 exec, %[s1], %[s0]\n\t" \
  "ds_write_b8_d16_hi v70, v230\n\t" \
  "s_mov_b64 exec, -1\n\t" \
  "v_add_u32 v72, v68, v67\n\t" \
  "v_add_u32 v76, v72, v66\n\t" \
  "v_and_b32 v80, -4, v72\n\t" \
  "ds_read2_b32 v[88:89], v80 offset1:1\n\t" \
  "v_and_b32 v73, 0x1010101, v64\n\t" \
  "v_bcnt_u32_b32 v73, v73, v72\n\t" \
  "v_and_b32 v81, -4, v73\n\t" \
  "ds_read2_b32 v[90:91], v81 offset1:1\n\t" \
  "v_and_b32 v74, 0x5050505, v64\n\t" \
  "v_bcnt_u32_b32 v74, v74, v72\n\t" \
  "v_and_b32 v82, -4, v74\n\t" \
  "ds_read2_b32 v[92:93], v82 offset1:1\n\t" \
  "v_and_b32 v75, 0x15151515, v64\n\t" \
  "v_bcnt_u32_b32 v75, v75, v72\n\t" \
  "v_and_b32 v83, -4, v75\n\t" \
  "ds_read2_b32 v[94:95], v83 offset1:1\n\t" \
  "v_and_b32 v84, -4, v76\n\t" \
  "ds_read2_b32 v[96:97], v84 offset1:1\n\t" \
  "v_and_b32 v77, 0x1010101, v65\n\t" \
  "v_bcnt_u32_b32 v77, v77, v76\n\t" \
  "v_and_b32 v85, -4, v77\n\t" \
  "ds_read2_b32 v[98:99], v85 offset1:1\n\t" \
  "v_and_b32 v78, 0x5050505, v65\n\t" \
  "v_bcnt_u32_b32 v78, v78, v76\n\t" \
  "v_and_b32 v86, -4, v78\n\t" \
  "ds_read2_b32 v[100:101], v86 offset1:1\n\t" \
  "v_and_b32 v79, 0x15151515, v65\n\t" \
  "v_bcnt_u32_b32 v79, v79, v76\n\t" \
  "v_and_b32 v87, -4, v79\n\t" \
  "ds_read2_b32 v[102:103], v87 offset1:1\n\t" \
  "s_lshr_b32 %[s0], %[up], 6\n\t" \
  "v_lshl_or_b32 v120, %[s0], %[fsh], %[vf]\n\t" \
  "s_waitcnt vmcnt(5)\n\t" \
  "s_lshl_b64 exec, 1, %[up]\n\t" \
  "global_store_dword v120, %[vcall], %[rdy] sc0 sc1\n\t" \
  "s_mov_b64 exec, -1\n\t" \
  "v_lshlrev_b32 v70, 8, v64\n\t" \
  "v_lshl_add_u32 v69, v64, 1, v64\n\t" \
  "v_lshl_add_u32 v71, v70, 8, v70\n\t" \
  "v_lshl_add_u32 v71, v70, 16, v71\n\t" \
  "v_bfi_b32 v71, v69, v71, v228\n\t" \
  "v_lshlrev_b32 v70, 2, v64\n\t" \
  "v_bfi_b32 v70, %[k33], v71, v70\n\t" \
  "v_lshrrev_b32 v71, 2, v71\n\t" \
  "v_bfi_b32 v71, %[k33], v71, v64\n\t" \
  "v_and_b32 v80, 0x7070707, v70\n\t" \
  "v_and_b32 v81, 0x7070707, v71\n\t" \
  "v_lshrrev_b32 v70, 4, v70\n\t" \
  "v_lshrrev_b32 v71, 4, v71\n\t" \
  "v_and_b32 v82, 0x7070707, v70\n\t" \
  "v_and_b32 v83, 0x7070707, v71\n\t" \
  "v_lshlrev_b32 v70, 8, v65\n\t" \
  "v_lshl_add_u32 v69, v65, 1, v65\n\t" \
  "v_lshl_add_u32 v71, v70, 8, v70\n\t" \
  "v_lshl_add_u32 v71, v70, 16, v71\n\t" \
  "v_bfi_b32 v71, v69, v71, v229\n\t" \
  "v_lshlrev_b32 v70, 2, v65\n\t" \
  "v_bfi_b32 v70, %[k33], v71, v70\n\t" \
  "v_lshrrev_b32 v71, 2, v71\n\t" \
  "v_bfi_b32 v71, %[k33], v71, v65\n\t" \
  "v_and_b32 v84, 0x7070707, v70\n\t" \
  "v_and_b32 v85, 0x7070707, v71\n\t" \
  "v_lshrrev_b32 v70, 4, v70\n\t" \
  "v_lshrrev_b32 v71, 4, v71\n\t" \
  "v_and_b32 v86, 0x7070707, v70\n\t" \
  "v_and_b32 v87, 0x7070707, v71\n\t" \
  "s_waitcnt lgkmcnt(0)\n\t" \
  "v_alignbyte_b32 v69, v89, v88, v72\n\t" \
  "v_perm_b32 v69, v69, %[lp], v80\n\t" \
  "v_bfi_b32 v70, %[k80], v69, v220\n\t" \
  "v_bfi_b32 v71, %[k80], v220, v69\n\t" \
  "v_perm_b32 v104, v71, v70, %[kp0]\n\t" \
  "v_perm_b32 v105, v71, v70, %[kp1]\n\t" \
  "v_alignbyte_b32 v69, v91, v90, v73\n\t" \
  "v_perm_b32 v69, v69, %[lp], v81\n\t" \
  "v_bfi_b32 v70, %[k80], v69, v221\n\t" \
  "v_bfi_b32 v71, %[k80], v221, v69\n\t" \
  "v_perm_b32 v106, v71, v70, %[kp0]\n\t" \
  "v_perm_b32 v107, v71, v70, %[kp1]\n\t" \
  "global_store_dwordx4 %[o1], v[104:107], %[xb] offset:0 sc0 sc1\n\t" \
  "v_alignbyte_b32 v69, v93, v92, v74\n\t" \
  "v_perm_b32 v69, v69, %[lp], v82\n\t" \
  "v_bfi_b32 v70, %[k80], v69, v222\n\t" \
  "v_bfi_b32 v71, %[k80], v222, v69\n\t" \
  "v_perm_b32 v108, v71, v70, %[kp0]\n\t" \
  "v_perm_b32 v109, v71, v70, %[kp1]\n\t" \
  "v_alignbyte_b32 v69, v95, v94, v75\n\t" \
  "v_perm_b32 v69, v69, %[lp], v83\n\t" \
  "v_bfi_b32 v70, %[k80], v69, v223\n\t" \
  "v_bfi_b32 v71, %[k80], v223, v69\n\t" \
  "v_perm_b32 v110, v71, v70, %[kp0]\n\t" \
  "v_perm_b32 v111, v71, v70, %[kp1]\n\t" \
  "global_store_dwordx4 %[o1], v[108:111], %[xb] offset:1024 sc0 sc1\n\t" \
  "v_alignbyte_b32 v69, v97, v96, v76\n\t" \
  "v_perm_b32 v69, v69, %[lp], v84\n\t" \
  "v_bfi_b32 v70, %[k80], v69, v224\n\t" \
  "v_bfi_b32 v71, %[k80], v224, v69\n\t" \
  "v_perm_b32 v112, v71, v70, %[kp0]\n\t" \
  "v_perm_b32 v113, v71, v70, %[kp1]\n\t" \
  "v_alignbyte_b32 v69, v99, v98, v77\n\t" \
  "v_perm_b32 v69, v69, %[lp], v85\n\t" \
  "v_bfi_b32 v70, %[k80], v69, v225\n\t" \
  "v_bfi_b32 v71, %[k80], v225, v69\n\t" \
  "v_perm_b32 v114, v71, v70, %[kp0]\n\t" \
  "v_perm_b32 v115, v71, v70, %[kp1]\n\t" \
  "global_store_dwordx4 %[o1], v[112:115], %[xb] offset:2048 sc0 sc1\n\t" \
  "v_alignbyte_b32 v69, v101, v100, v78\n\t" \
  "v_perm_b32 v69, v69, %[lp], v86\n\t" \
  "v_bfi_b32 v70, %[k80], v69, v226\n\t" \
  "v_bfi_b32 v71, %[k80], v226, v69\n\t" \
  "v_perm_b32 v116, v71, v70, %[kp0]\n\t" \
  "v_perm_b32 v117, v71, v70, %[kp1]\n\t" \
  "v_alignbyte_b32 v69, v103, v102, v79\n\t" \
  "v_perm_b32 v69, v69, %[lp], v87\n\t" \
  "v_bfi_b32 v70, %[k80], v69, v227\n\t" \
  "v_bfi_b32 v71, %[k80], v227, v69\n\t" \
  "v_perm_b32 v118, v71, v70, %[kp0]\n\t" \
  "v_perm_b32 v119, v71, v70, %[kp1]\n\t" \
  "global_store_dwordx4 %[o1], v[116:119], %[xb] offset:3072 sc0 sc1\n\t" \
  "s_waitcnt vmcnt(8)\n\t" \
  "s_mov_b32 %[up], %[u1]\n\t" \
  "v_readfirstlane_b32 %[u1], v254\n\t" \
  "s_min_u32 %[u1], %[u1], " TK_MAX "\n\t" \
  "" TK_ADD " %[u1], %[u1], %[nws]\n\t" \
  "s_sub_u32 %[s0], %[u1], %[wb]\n\t" \
  "v_readlane_b32 %[h1], v252, %[s0]\n\t" \
  "v_readlane_b32 %[hc], v253, %[s0]\n\t" \
  "s_add_u32 %[wb], %[u1], 1\n\t" \
  "v_lshl_add_u32 v71, %[wb], 3, %[v8]\n\t" \
  "s_or_b32 %[s0], %[u1], %[tag]\n\t" \
  "global_load_dwordx2 v[252:253], v71, %[hdr] sc0 sc1\n\t" \
  "s_mov_b64 exec, 1\n\t" \
  "global_atomic_add v254, %[v16], %[one], %[tkt] sc0 sc1\n\t" \
  "s_mov_b64 exec, -1\n\t" \
  "v_lshl_add_u32 %[o1], %[u1], 12, %[v16]\n\t" \
  "v_lshl_add_u32 %[o12], %[u1], 12, %[v12]\n\t" \
  "s_and_b32 %[s1], %[h1], 0x830000ff\n\t" \
  "s_cmp_eq_u32 %[hc], %[s0]\n\t" \
  "s_cselect_b32 %[s1], %[s1], 0\n\t" \
  "s_cmp_eq_u32 %[s1], %[pfk]\n\t" \
  "s_cbranch_scc1 Lf0_%=\n\t" \
  "s_branch Lf3_%=\n\t" \
  "Lf2_%=:\n\t" \
  "s_waitcnt vmcnt(7)\n\t" \
  "v_mov_b32 v236, v220\n\t" \
  "v_mov_b32 v237, v221\n\t" \
  "v_mov_b32 v238, v222\n\t" \
  "v_mov_b32 v239, v223\n\t" \
  "v_mov_b32 v240, v224\n\t" \
  "v_mov_b32 v241, v225\n\t" \
  "v_mov_b32 v242, v226\n\t" \
  "v_mov_b32 v243, v227\n\t" \
  "v_mov_b32 v244, v228\n\t" \
  "v_mov_b32 v245, v229\n\t" \
  "v_mov_b32 v246, v230\n\t" \
  "v_swap_b32 %[o0], %[o1]\n\t" \
  "s_mov_b32 %[s1], %[h0]\n\t" \
  "s_mov_b32 %[h0], %[h1]\n\t" \
  "s_mov_b32 %[h1], %[s1]\n\t" \
  "s_mov_b32 %[s1], %[u0]\n\t" \
  "s_mov_b32 %[u0], %[u1]\n\t" \
  "s_mov_b32 %[u1], %[s1]\n\t" \
  "Lf3_%=:\n\t" \
  ""
#define FAST_CLOB "v64", "v65", "v66", "v67", "v68", "v69", "v70", "v71", "v72", "v73", "v74", "v75", "v76", "v77", "v78", "v79", "v80", "v81", "v82", "v83", "v84", "v85", "v86", "v87", "v88", "v89", "v90", "v91", "v92", "v93", "v94", "v95", "v96", "v97", "v98", "v99", "v100", "v101", "v102", "v103", "v104", "v105", "v106", "v107", "v108", "v109", "v110", "v111", "v112", "v113", "v114", "v115", "v116", "v117", "v118", "v119", "v120", "v220", "v221", "v222", "v223", "v224", "v225", "v226", "v227", "v228", "v229", "v230"

#define LAS __attribute__((address_space(3)))
typedef unsigned u32;
typedef unsigned short u16;
typedef unsigned long long u64;
typedef u32 u32x4 __attribute__((ext_vector_type(4)));
typedef u32 u32x2 __attribute__((ext_vector_type(2)));
typedef float f32x4 __attribute__((ext_vector_type(4)));
typedef float f32x16 __attribute__((ext_vector_type(16)));
typedef float f32x32 __attribute__((ext_vector_type(32)));
typedef __bf16 bf16x2 __attribute__((ext_vector_type(2)));
typedef __bf16 bf16x8 __attribute__((ext_vector_type(8)));
typedef float f32x8 __attribute__((ext_vector_type(8)));
typedef float f32x2 __attribute__((ext_vector_type(2)));
typedef unsigned short u16x2 __attribute__((ext_vector_type(2)));

#define V_PF0 (3 * 64 * 4096)
#define V_ELAG 1
#define V_FAST 1
#define V_FL 8
#define V_CS 32
#define V_NS 0
#define TK_ADD "s_add_u32"
#define TK_MAX "0x400"
#define V_WIN "3"
#define V_MA 1
#define V_PFL 16
#define V_HW 39u
#define V_NPW 133
namespace {
constexpr int NR = 8, KD = 4096, NL = 2048, ML = 512;
constexpr int NT = 64;
constexpr int TILE = 16384, ASLOT = 20480, WSLOT = 16384;
constexpr int A_OFF = 0, W_OFF = 3 * ASLOT, C_OFF = W_OFF + 6 * WSLOT, LDS_BYTES = C_OFF + 64;
constexpr int C_BAD = 0, C_HDR = 16;
constexpr int PU_LDS = 16384;
constexpr size_t WS_CNT = 262144;
constexpr size_t WS_TMO = 4096, WS_DUMMY = 393216;
constexpr size_t WS_TKT = 8192;
constexpr size_t WS_RDY = 1u << 20;
constexpr u32 FS = 1u << V_FL;
constexpr size_t WS_HDR = 65536;
constexpr size_t WS_PK = 2u << 20;
constexpr size_t WS_X = 16u << 20;
constexpr size_t XPAR = (size_t)8 * 1024 * 4096;
constexpr u32 TP = 2, U0 = 16 * TP;
constexpr size_t WS_NEED = WS_X + XPAR + (size_t)2 * 8 * U0 * 4096;
constexpr u64 DEADLINE = 50000000ull;

struct Params {
  const char* x; const char* w; char* out; char* my; char* peer[NR - 1]; int rank; int pad;
};

__device__ __forceinline__ u64 rt() { return __builtin_amdgcn_s_memrealtime(); }
__device__ __forceinline__ bool late(u64& t0) { const u64 t = rt(); if (!t0) t0 = t; return t - t0 > DEADLINE; }
__device__ __forceinline__ u32 rfl(u32 v) { return __builtin_amdgcn_readfirstlane(v); }
__device__ __forceinline__ u32 rdl(u32 v, u32 l) { return __builtin_amdgcn_readlane(v, l); }
__device__ __forceinline__ u32x4 mk_rsrc(const void* p, u32 bytes) {
  u64 a = (u64)p; u32x4 r; r.x = rfl((u32)a); r.y = rfl((u32)(a >> 32)) & 0xffffu; r.z = bytes; r.w = 0x00020000u; return r;
}
template <int SYS> __device__ __forceinline__ void glds16x4(u32 lds, u32 v0, u32 v1, u32 v2, u32 v3, u32x4 rsrc, u32 soff) {
  if (SYS) asm volatile("s_mov_b32 m0, %0\n\ts_nop 0\n\tbuffer_load_dwordx4 %1, %5, %6 offen sc0 sc1 lds" "\n\tbuffer_load_dwordx4 %2, %5, %6 offen offset:1024 sc0 sc1 lds" "\n\t"
                        "buffer_load_dwordx4 %3, %5, %6 offen offset:2048 sc0 sc1 lds" "\n\tbuffer_load_dwordx4 %4, %5, %6 offen offset:3072 sc0 sc1 lds"
                        :: "s"(lds), "v"(v0), "v"(v1), "v"(v2), "v"(v3), "s"(rsrc), "s"(soff) : "memory");
  else     asm volatile("s_mov_b32 m0, %0\n\ts_nop 0\n\tbuffer_load_dwordx4 %1, %5, %6 offen lds" "\n\tbuffer_load_dwordx4 %2, %5, %6 offen offset:1024 lds" "\n\t"
                        "buffer_load_dwordx4 %3, %5, %6 offen offset:2048 lds" "\n\tbuffer_load_dwordx4 %4, %5, %6 offen offset:3072 lds"
                        :: "s"(lds), "v"(v0), "v"(v1), "v"(v2), "v"(v3), "s"(rsrc), "s"(soff) : "memory");
}
__device__ __forceinline__ void glds16x2(u32 lds, u32 v0, u32 v1, u32x4 rsrc, u32 soff) {
  asm volatile("s_mov_b32 m0, %0\n\ts_nop 0\n\tbuffer_load_dwordx4 %1, %3, %4 offen lds\n\tbuffer_load_dwordx4 %2, %3, %4 offen offset:1024 lds"
               :: "s"(lds), "v"(v0), "v"(v1), "s"(rsrc), "s"(soff) : "memory");
}
__device__ __forceinline__ u32x4 lds_rd4(u32 addr) { u32x4 r; asm volatile("ds_read_b128 %0, %1\n\ts_waitcnt lgkmcnt(0)" : "=&v"(r) : "v"(addr) : "memory"); return r; }
__device__ __forceinline__ u32 lds_rd1(u32 addr) { u32 r; asm volatile("ds_read_b32 %0, %1\n\ts_waitcnt lgkmcnt(0)" : "=&v"(r) : "v"(addr) : "memory"); return r; }
__device__ __forceinline__ void lds_wr1(u32 addr, u32 v) { asm volatile("ds_write_b32 %0, %1" :: "v"(addr), "v"(v) : "memory"); }
#define WAIT_VM(n) asm volatile("s_waitcnt vmcnt(" #n ")" ::: "memory")
__device__ __forceinline__ void st16_sys(char* p, u32x4 v) { asm volatile("global_store_dwordx4 %0, %1, off sc0 sc1\n\ts_nop 1" :: "v"(p), "v"(v) : "memory"); }
__device__ __forceinline__ u32 pk_bf16(float a, float b) { f32x2 f = {a, b}; bf16x2 h = __builtin_convertvector(f, bf16x2); return __builtin_bit_cast(u32, h); }
#define CBAR() asm volatile("" ::: "memory")
__device__ __forceinline__ u32 perm(u32 a, u32 b, u32 s) { return __builtin_amdgcn_perm(a, b, s); }
__device__ __forceinline__ u32 bfi(u32 m, u32 a, u32 b) { return (a & m) | (b & ~m); }
__device__ __forceinline__ u32 wave_scan(u32 x) {
  x += __builtin_amdgcn_update_dpp(0u, x, 0x111, 0xf, 0xf, false);
  x += __builtin_amdgcn_update_dpp(0u, x, 0x112, 0xf, 0xf, false);
  x += __builtin_amdgcn_update_dpp(0u, x, 0x114, 0xf, 0xf, false);
  x += __builtin_amdgcn_update_dpp(0u, x, 0x118, 0xf, 0xf, false);
  x += __builtin_amdgcn_update_dpp(0u, x, 0x142, 0xa, 0xf, false);
  x += __builtin_amdgcn_update_dpp(0u, x, 0x143, 0xc, 0xf, false);
  return x;
}
__device__ __forceinline__ void st8_sys(char* p, u32x2 v) { asm volatile("global_store_dwordx2 %0, %1, off sc0 sc1\n\ts_nop 1" :: "v"(p), "v"(v) : "memory"); }
__device__ __forceinline__ void st4_sys(char* p, u32 v) { asm volatile("global_store_dword %0, %1, off sc0 sc1\n\ts_nop 1" :: "v"(p), "v"(v) : "memory"); }
template <int OFF> __device__ __forceinline__ void st16_s(char* base, u32 voff, u32x4 v) {
  asm volatile("global_store_dwordx4 %0, %1, %2 offset:%3 sc0 sc1\n\ts_nop 1" :: "v"(voff), "v"(v), "s"(base), "n"(OFF) : "memory"); }
constexpr u32 PK_SCR = 8320;
constexpr u32 UP_SCR = 2112;

template <int B> __device__ __forceinline__ LAS u16* add_byte(u32 base, u32 o) {
  u32 r;
  if (B == 0) asm("v_add_u32_sdwa %0, %1, %2 dst_sel:DWORD dst_unused:UNUSED_PAD src0_sel:DWORD src1_sel:BYTE_0" : "=v"(r) : "v"(base), "v"(o));
  if (B == 1) asm("v_add_u32_sdwa %0, %1, %2 dst_sel:DWORD dst_unused:UNUSED_PAD src0_sel:DWORD src1_sel:BYTE_1" : "=v"(r) : "v"(base), "v"(o));
  if (B == 2) asm("v_add_u32_sdwa %0, %1, %2 dst_sel:DWORD dst_unused:UNUSED_PAD src0_sel:DWORD src1_sel:BYTE_2" : "=v"(r) : "v"(base), "v"(o));
  if (B == 3) asm("v_add_u32_sdwa %0, %1, %2 dst_sel:DWORD dst_unused:UNUSED_PAD src0_sel:DWORD src1_sel:BYTE_3" : "=v"(r) : "v"(base), "v"(o));
  return (LAS u16*)(size_t)r;
}
__device__ __forceinline__ u32 pre3(u32 e) {
  u32 u = e << 8, t, x;
  asm("v_lshl_add_u32 %0, %1, 8, %1" : "=v"(t) : "v"(u));
  asm("v_lshl_add_u32 %0, %1, 16, %2" : "=v"(x) : "v"(u), "v"(t));
  return x;
}
__device__ __forceinline__ u32 pack_unit(const u32x4* c, char* pk, LAS char* scr, u32 lane) {
  u32 sum = 0, hi[8]; u32x4 B[2];
#pragma unroll
  for (int g = 0; g < 8; g += 2) { hi[g] = perm(c[g >> 1][1], c[g >> 1][0], 0x07050301u); sum = __builtin_amdgcn_sad_u8(hi[g] & 0x7f7f7f7fu, 0u, sum); }
  sum += __builtin_amdgcn_update_dpp(0u, sum, 0x111, 0xf, 0xf, false);
  sum += __builtin_amdgcn_update_dpp(0u, sum, 0x112, 0xf, 0xf, false);
  sum += __builtin_amdgcn_update_dpp(0u, sum, 0x114, 0xf, 0xf, false);
  sum += __builtin_amdgcn_update_dpp(0u, sum, 0x118, 0xf, 0xf, false);
#pragma unroll
  for (int g = 0; g < 8; ++g) {
    const u32 a = c[g >> 1][2 * (g & 1)], b = c[g >> 1][2 * (g & 1) + 1];
    if (g & 1) hi[g] = perm(b, a, 0x07050301u);
    B[g >> 2][g & 3] = bfi(0x80808080u, hi[g], perm(b, a, 0x06040200u));
  }
  const u32 F = (rdl(sum, 15) + rdl(sum, 47) + 128u) >> 8;
  const u32 kf = ((256u - F) & 255u) * 0x01000100u;
  st16_sys(pk + lane * 16, B[0]); st16_sys(pk + 1024 + lane * 16, B[1]);
  u32 M[16], P[8]; u32x2 C, E;
#pragma unroll
  for (int i = 0; i < 16; ++i) asm("v_pk_mad_u16 %0, %1, %2, %3" : "=v"(M[i]) : "v"(c[i >> 2][i & 3]), "v"(0x00020002u), "s"(kf));
#pragma unroll
  for (int g = 0; g < 8; ++g) {
    const u16x2 lim = {0x0300, 0x0300};
    const u32 p0 = __builtin_bit_cast(u32, __builtin_elementwise_min(__builtin_bit_cast(u16x2, M[2 * g]), lim));
    const u32 p1 = __builtin_bit_cast(u32, __builtin_elementwise_min(__builtin_bit_cast(u16x2, M[2 * g + 1]), lim));
    P[g] = perm(p1, p0, 0x07050301u);
  }
#pragma unroll
  for (int d = 0; d < 2; ++d) {
    C[d] = (P[4 * d] | (P[4 * d + 1] << 2)) | ((P[4 * d + 2] | (P[4 * d + 3] << 2)) << 4);
    E[d] = C[d] & (C[d] >> 1) & 0x55555555u;
  }
  const u32 tot = __builtin_popcount(E[0]) + __builtin_popcount(E[1]);
  const u32 inc = wave_scan(tot);
  u32 o[8], cum[8];
#pragma unroll
  for (int d = 0; d < 2; ++d) {
    const u32 X = pre3(E[d]), nE = E[d] ^ 0x55555555u;
    const u32 A0 = bfi(0xccccccccu, X << 2, nE << 1), A1 = bfi(0xccccccccu, X, nE >> 1);
    o[4 * d] = A0 & 0x0f0f0f0fu; o[4 * d + 1] = A1 & 0x0f0f0f0fu; o[4 * d + 2] = (A0 >> 4) & 0x0f0f0f0fu; o[4 * d + 3] = (A1 >> 4) & 0x0f0f0f0fu;
  }
  cum[0] = 0; cum[4] = __builtin_popcount(E[0]);
#pragma unroll
  for (int d = 0; d < 2; ++d) {
    cum[4 * d + 1] = cum[4 * d] + __builtin_popcount(E[d] & 0x01010101u); cum[4 * d + 2] = cum[4 * d] + __builtin_popcount(E[d] & 0x05050505u);
    cum[4 * d + 3] = cum[4 * d] + __builtin_popcount(E[d] & 0x15151515u);
  }
  st8_sys(pk + 2048 + lane * 12, C);
  const u32 n = rdl(inc, 63), ne = (n + 9u) / 10u;
  const u32 b0 = (u32)(size_t)scr + (inc - tot) * 4;
#pragma unroll
  for (int g = 0; g < 8; ++g) {
    const u32 bp = b0 + cum[g] * 4;
    *add_byte<0>(bp, o[g]) = (u16)M[2 * g]; *add_byte<1>(bp, o[g]) = (u16)(M[2 * g] >> 16);
    *add_byte<2>(bp, o[g]) = (u16)M[2 * g + 1]; *add_byte<3>(bp, o[g]) = (u16)(M[2 * g + 1] >> 16);
  }
  *(LAS u16*)(scr + (n + (lane & 15)) * 4) = (u16)0xff00u;
  CBAR();
  u32 nx = 0;
  for (u32 s0 = 0; s0 < min(ne, 128u); s0 += 64) {
    const u32 slot = s0 + lane;
    const LAS u32x2* rp = (const LAS u32x2*)(scr + slot * 40);
    const u32x2 d0 = rp[0], d1 = rp[1], d2 = rp[2], d3 = rp[3], d4 = rp[4];
    u32 s[3], k[3], df[3];
    s[0] = perm(d0.y, d0.x, 0x0c0c0501u) | perm(d1.y, d1.x, 0x05010c0cu);
    s[1] = perm(d2.y, d2.x, 0x0c0c0501u) | perm(d3.y, d3.x, 0x05010c0cu);
    s[2] = perm(d4.y, d4.x, 0x0d050d01u);
#pragma unroll
    for (int r = 0; r < 3; ++r) { k[r] = s[r] & (s[r] >> 3) & 0x07070707u; df[r] = perm(0xfffefdfcu, 0xfbfaf903u, k[r]) ^ s[r]; }
    const u32 v = k[0] | (k[1] << 4) | (((k[2] & 0x00070007u) * 0x248u) & 0x08880888u);
    if (slot < ne) st4_sys(pk + (s0 ? 2816 + lane * 4 : 2056 + lane * 12), v);
    for (;;) {
      const bool has = slot < ne && (df[0] | df[1] | df[2]) != 0u;
      const u64 em = __ballot(has);
      if (!em || nx > 32u) break;
      const bool f0 = df[0] != 0u, f1 = df[1] != 0u;
      const u32 dd = f0 ? df[0] : f1 ? df[1] : df[2], ss = f0 ? s[0] : f1 ? s[1] : s[2];
      const u32 b8 = __builtin_ctz(dd | 0x80000000u) & 24u, e = (ss >> b8) + F, keep = ~(255u << b8);
      const u32 ent = (10u * slot + (f0 ? b8 >> 3 : f1 ? 4u + (b8 >> 3) : 8u + (b8 >> 4))) | ((e & 0xfeu) << 15) | ((e & 1u) << 23);
      const u32 kx = ne + nx + __builtin_amdgcn_mbcnt_hi((u32)(em >> 32), __builtin_amdgcn_mbcnt_lo((u32)em, 0u));
      if (has && kx < 128u) st4_sys(pk + (kx < 64u ? 2056 + kx * 12 : 2560 + kx * 4), ent);
      df[2] &= (f0 || f1) ? ~0u : keep; df[1] &= f0 ? ~0u : keep; df[0] &= keep;
      nx += __builtin_popcountll(em);
    }
  }
  CBAR();
  if (ne + nx > 128u || nx > 32u) {
    WAIT_VM(0);
    st16_sys(pk + lane * 16, c[0]); st16_sys(pk + 1024 + lane * 16, c[1]); st16_sys(pk + 2048 + lane * 16, c[2]); st16_sys(pk + 3072 + lane * 16, c[3]);
    return 0x81000000u;
  }
  return F | (ne << 8) | (nx << 16) | (ne + nx > 64u ? 2u << 24 : 3u << 24);
}

__device__ __forceinline__ u32 exp_lut(u32 f4, u32 s) {
  const u32 v = ((f4 & 0x7f7f7f7fu) + (s & 0x7f7f7f7fu)) ^ ((f4 ^ s) & 0x80808080u);
  return ((v >> 1) & 0x7f7f7f7fu) | ((v << 7) & 0x80808080u);
}
__device__ __forceinline__ void unpack_unit(u32x4* c, u32 h, u32 lp, u32 l0, u32 l1, LAS char* scr, u32 lane) {
  const u32x4 B0 = c[0], B1 = c[1]; const u32x2 C = c[2].lo; const u32 V0 = c[2].z, V1 = c[2].w;
  const u32 ne = (h >> 8) & 255u, nx = (h >> 16) & 255u;
  auto expand = [&](u32 V, u32 slot) {
    const u16x2 mu = {0x124, 0x124};
    const u32 k2 = __builtin_bit_cast(u32, (u16x2)(__builtin_bit_cast(u16x2, (V >> 3) & 0x01110111u) * mu));
    const u32 r0 = perm(l1, l0, V & 0x07070707u), r1 = perm(l1, l0, (V >> 4) & 0x07070707u), r2 = perm(l1, l0, perm(k2, k2, 0x0c0c0301u) & 0x0707u);
    asm volatile("ds_write_b16 %0, %1\n\tds_write_b16_d16_hi %0, %1 offset:2\n\tds_write_b16 %0, %2 offset:4\n\tds_write_b16_d16_hi %0, %2 offset:6\n\tds_write_b16 %0, %3 offset:8"
                 :: "v"((u32)(size_t)(scr + slot * 10)), "v"(r0), "v"(r1), "v"(r2) : "memory");
  };
  u32x2 E;
#pragma unroll
  for (int d = 0; d < 2; ++d) E[d] = C[d] & (C[d] >> 1) & 0x55555555u;
  const u32 t0 = __builtin_popcount(E[0]), tot = t0 + __builtin_popcount(E[1]);
  const u32 inc = wave_scan(tot);
  expand(V0, lane);
  if (__builtin_expect(ne + nx > 64u, 0)) {
    expand(V1, lane + 64);
    CBAR();
    *(LAS unsigned char*)(scr + (lane + 64u - ne < nx ? V1 & 2047u : 2048u + lane)) = (unsigned char)(V1 >> 16);
  }
  CBAR();
  *(LAS unsigned char*)(scr + (lane - ne < nx ? V0 & 2047u : 2048u + lane)) = (unsigned char)(V0 >> 16);
  CBAR();
  u32 ad[8], es[8], sel[8];
  ad[0] = inc - tot + (u32)(size_t)scr; ad[4] = ad[0] + t0;
#pragma unroll
  for (int d = 0; d < 2; ++d) {
    ad[4 * d + 1] = ad[4 * d] + __builtin_popcount(E[d] & 0x01010101u); ad[4 * d + 2] = ad[4 * d] + __builtin_popcount(E[d] & 0x05050505u);
    ad[4 * d + 3] = ad[4 * d] + __builtin_popcount(E[d] & 0x15151515u);
  }
  u32x2 q[8];
  asm volatile("ds_read2_b32 %0, %8 offset1:1\n\tds_read2_b32 %1, %9 offset1:1\n\tds_read2_b32 %2, %10 offset1:1\n\tds_read2_b32 %3, %11 offset1:1\n\t"
               "ds_read2_b32 %4, %12 offset1:1\n\tds_read2_b32 %5, %13 offset1:1\n\tds_read2_b32 %6, %14 offset1:1\n\tds_read2_b32 %7, %15 offset1:1"
               : "=&v"(q[0]), "=&v"(q[1]), "=&v"(q[2]), "=&v"(q[3]), "=&v"(q[4]), "=&v"(q[5]), "=&v"(q[6]), "=&v"(q[7])
               : "v"(ad[0] & ~3u), "v"(ad[1] & ~3u), "v"(ad[2] & ~3u), "v"(ad[3] & ~3u), "v"(ad[4] & ~3u), "v"(ad[5] & ~3u), "v"(ad[6] & ~3u), "v"(ad[7] & ~3u) : "memory");
  u32x2 Cv = C;
  asm volatile("" : "+v"(Cv), "+v"(E));
#pragma unroll
  for (int d = 0; d < 2; ++d) {
    const u32 Lw = bfi(E[d] * 3u, pre3(E[d]), Cv[d]);
    u32 A0, A1;
    asm("v_bfi_b32 %0, %1, %2, %3" : "=v"(A0) : "s"(0x33333333u), "v"(Lw), "v"(E[d] << 2));
    asm("v_bfi_b32 %0, %1, %2, %3" : "=v"(A1) : "s"(0x33333333u), "v"(Lw >> 2), "v"(E[d]));
    sel[4 * d] = A0 & 0x07070707u; sel[4 * d + 1] = A1 & 0x07070707u; sel[4 * d + 2] = (A0 >> 4) & 0x07070707u; sel[4 * d + 3] = (A1 >> 4) & 0x07070707u;
  }
  asm volatile("" : "+v"(sel[0]), "+v"(sel[1]), "+v"(sel[2]), "+v"(sel[3]), "+v"(sel[4]), "+v"(sel[5]), "+v"(sel[6]), "+v"(sel[7]));
  asm volatile("s_waitcnt lgkmcnt(0)" : "+v"(q[0]), "+v"(q[1]), "+v"(q[2]), "+v"(q[3]), "+v"(q[4]), "+v"(q[5]), "+v"(q[6]), "+v"(q[7]) :: "memory");
#pragma unroll
  for (int g = 0; g < 8; ++g) es[g] = __builtin_amdgcn_alignbyte(q[g].y, q[g].x, ad[g]);
#pragma unroll
  for (int g = 0; g < 8; ++g) {
    const u32 R = perm(es[g], lp, sel[g]), b = g < 4 ? B0[g & 3] : B1[g & 3];
    const u32 lo = bfi(0x80808080u, R, b), hi = bfi(0x80808080u, b, R);
    c[g >> 1][2 * (g & 1)] = perm(hi, lo, 0x05010400u); c[g >> 1][2 * (g & 1) + 1] = perm(hi, lo, 0x07030602u);
  }
}
__device__ __forceinline__ u32 load_unit(const char* x, u32 u, u32 lane, u32x4* c, const u32* cnt) {
  u32 r;
  const char* sp = x + (size_t)((u & 15) * 32 + (lane >> 3)) * (KD * 2) + (u >> 4) * 128;
  const u32 c0 = 16 * ((lane & 7) ^ (lane >> 4)), c1 = c0 ^ 64u;
  asm volatile("global_load_dwordx4 %0, %5, off\n\tglobal_load_dwordx4 %1, %6, off\n\tglobal_load_dwordx4 %2, %7, off\n\tglobal_load_dwordx4 %3, %8, off\n\t"
               "global_load_dword %4, %9, off sc1\n\ts_waitcnt vmcnt(0)"
               : "=&v"(c[0]), "=&v"(c[1]), "=&v"(c[2]), "=&v"(c[3]), "=&v"(r)
               : "v"(sp + c0), "v"(sp + 8 * KD * 2 + c1), "v"(sp + 16 * KD * 2 + c0), "v"(sp + 24 * KD * 2 + c1), "v"(cnt) : "memory");
  return r;
}
__device__ __forceinline__ f32x4 mma(f32x4 w, f32x4 x, f32x4 c) { return __builtin_amdgcn_mfma_f32_16x16x32_bf16(__builtin_bit_cast(bf16x8, w), __builtin_bit_cast(bf16x8, x), c, 0, 0, 0); }
__device__ __forceinline__ void store_tile(f32x4* a, const f32x4* wf, const f32x4* xq, LAS char* w0, LAS char* w1, LAS char* rd, int last, char* out, u32 o, u32 olast) {
  u32x4 r[9][2];
  auto fin = [&](int m) {
#pragma unroll
    for (int n = 0; n < 4; ++n) a[4 * m + n] = mma(wf[n], xq[m], a[4 * m + n]);
    asm volatile("" :: "v"(xq[m])); };
#pragma unroll
  for (int m = 0; m < V_MA; ++m) fin(m);
#pragma unroll
  for (int m = 0; m < 9 + V_ELAG; ++m) {
    if (m + V_MA < 9) fin(m + V_MA);
    __builtin_amdgcn_sched_barrier(0);
    if (m < 9) {
      const int off = m == 8 ? last : m * 2048;
#pragma unroll
      for (int h = 0; h < 2; ++h) {
        const f32x4 p = a[4 * m + 2 * h], q = a[4 * m + 2 * h + 1];
        u32 d0 = pk_bf16(p[0], p[1]), d1 = pk_bf16(p[2], p[3]), e0 = pk_bf16(q[0], q[1]), e1 = pk_bf16(q[2], q[3]);
        auto r0 = __builtin_amdgcn_permlane16_swap(d0, e0, false, false);
        auto r1 = __builtin_amdgcn_permlane16_swap(d1, e1, false, false);
        *(LAS u32x4*)((h ? w1 : w0) + off) = (u32x4){r0[0], r1[0], r0[1], r1[1]};
      }
      CBAR();
      r[m][0] = *(LAS u32x4*)(rd + off); r[m][1] = *(LAS u32x4*)(rd + off + 1024);
    }
    if (m >= V_ELAG) {
      const int k = m - V_ELAG;
#pragma unroll
      for (int h = 0; h < 2; ++h)
        asm volatile("global_store_dwordx4 %0, %1, %2 sc0 sc1\n\ts_nop 1" :: "v"((k == 8 ? olast : o + k * 16 * NL * 2) + h * 8 * NL * 2), "v"(r[k][h]), "s"(out) : "memory");
    } else CBAR();
  }
}
}

__global__ void __launch_bounds__(512, 2) ag_gemm_kernel(Params P) {
  extern __shared__ __attribute__((aligned(16))) char smem[];
  const u32 lbase = (u32)(size_t)(LAS char*)smem;
  const u32 cbase = lbase + C_OFF;
  const u32 tid = threadIdx.x, lane = tid & 63;
  const u32 wave = rfl(tid >> 6);
  asm volatile("" :: "s"(P.x), "s"(P.w), "s"(P.out), "s"(P.my), "s"(P.rank));
  const u32 rank = (u32)P.rank & 7u;
  const u32 xcd = blockIdx.x & 7, loc = blockIdx.x >> 3;
  const bool is_main = loc < 28, extra = loc < 16;
  const u32 cb = (xcd & 1) * 4 + (loc & 3);
  const u32 xf = extra ? loc >> 2 : 0u, xrow = 448 + (xcd >> 1) * 16;
  const u32 rbi = (xcd >> 1) * 7 + (loc >> 2);
  const u32 rb = ((rank + 1 + (loc >> 2)) & 7) * 4 + (((xcd >> 1) + (loc >> 2)) & 3);
  char* myws = P.my;
  u32* cnt = (u32*)(myws + WS_CNT) + blockIdx.x * V_CS + wave;
  u32* tmo = (u32*)(myws + WS_TMO);

  const bool packer = is_main ? wave < 4 : wave == 7;
  const u32 pc = ((loc - 28) * 8 + xcd) & 31u;
  const u32 si = (wave + pc) % 7u;
  const u32 wi = (pc + 5 * si) & 31u;
  char* const pws = P.peer[si];
  u32 call, hreg = 0;
  char* hp = nullptr;
  if (wave == (is_main ? 6u : 7u)) hp = P.peer[min(lane >> 2, 6u)] + WS_HDR + ((size_t)rank * 1024 + blockIdx.x * 4 + (lane & 3u)) * 8;
  if (packer) {
    u32x4 c[4];
    u32 u = blockIdx.x * 4 + (is_main ? wave : 0u);
    call = rfl(load_unit(P.x, u, lane, c, cnt)) + 1u;
    for (;;) {
      const size_t ui = (size_t)(call & 1u) * 1024 + u;
      const u32 h = pack_unit(c, myws + WS_PK + ui * 4096, (LAS char*)smem + wave * PU_LDS, lane);
      if (is_main) { *(LAS u32*)(smem + C_OFF + C_HDR + wave * 4) = h; break; }
      if ((lane & 3u) == (u & 3u)) hreg = h;
      if ((++u & 3u) == 0u) break;
      load_unit(P.x, u, lane, c, cnt);
    }
    asm volatile("s_waitcnt vmcnt(0) lgkmcnt(0)" ::: "memory");
  } else if (!is_main) {
    u32x4 c[4];
    call = rfl(load_unit(P.x, wi, lane, c, cnt)) + 1u;
    char* d = pws + WS_X + XPAR + (size_t)(((call & 1u) * 8 + rank) * U0 + wi) * 4096;
    st16_s<0>(d, lane * 16, c[0]); st16_s<1024>(d, lane * 16, c[1]); st16_s<2048>(d, lane * 16, c[2]); st16_s<3072>(d, lane * 16, c[3]);
  } else call = rfl(__hip_atomic_load(cnt, __ATOMIC_RELAXED, __HIP_MEMORY_SCOPE_AGENT)) + 1u;
  const u32 par = call & 1u;
  if (is_main) {
    if (wave == 6) {
      const u32 c1 = call - 1u;
      *(LAS u32*)(smem + C_OFF + C_BAD) = c1;
      asm volatile("s_waitcnt lgkmcnt(0)" : "+v"(hp) :: "memory");
    }
    __builtin_amdgcn_s_barrier();
    if (wave == 6) hreg = lds_rd1(cbase + C_HDR + (lane & 3u) * 4);
  }
  if (wave == (is_main ? 6u : 7u)) {
    if (lane < 28) st8_sys(hp + (size_t)par * 8 * 1024 * 8, (u32x2){hreg, call << 12 | blockIdx.x * 4 | (lane & 3u)});
  }
  if (lane == 0) __hip_atomic_store(cnt, call, __ATOMIC_RELAXED, __HIP_MEMORY_SCOPE_AGENT);

  if (!is_main) {
    if (wave == 7) return;
    if (pc < 4u) __hip_atomic_store((u32*)(myws + WS_TKT) + (((par ^ 1u) * 8 + si) * 4 + pc) * 64, 0u, __ATOMIC_RELAXED, __HIP_MEMORY_SCOPE_AGENT);
    const u32 gi = wave * 32 + pc, pi = rfl(gi % 7u);
    const u32 src = (rank + 1 + pi) & 7u;
    const u32 sub = rfl(gi / 7u) & ((1u << V_NS) - 1u);
    const u32 nws = rfl((V_NPW + 6 - pi) / 7u) + U0 + sub;
    const char* const sws = P.peer[pi];
    u32* tkt = (u32*)(myws + WS_TKT) + ((par * 8 + pi) * 4 + sub) * 64;
    const char* hdr = myws + WS_HDR + ((size_t)par * 8 + src) * 1024 * 8;
    const char* pk = sws + WS_PK + (size_t)par * 1024 * 4096;
    char* xb = myws + WS_X + (size_t)src * 1024 * 4096;
    char* const rdy = myws + WS_RDY + src * 256 * FS;
    char* fb = pws + WS_RDY + rank * 256 * FS;
    u32 up = wi, u = 0;
    u64 t0 = 0;
    u32 un = gi < V_NPW ? rfl(gi / 7u) + U0 : 1024u;
    const u32 v16 = lane * 16, v8 = min(lane, V_HW) * 8, v12 = lane * 12, vf = (lane & 12) * 16 * FS + (lane >> 4) * FS + (lane & 3) * 4, tag = call << 12;
    u32 wb = 0, wmin = 0;
#define LAND_REGS "v236", "v237", "v238", "v239", "v240", "v241", "v242", "v243", "v244", "v245", "v246", "v247", "v248", "v249", "v250", "v251", "v252", "v253", "v254"
    u32 kind = 0, o16 = 0, o12 = 0, od = 0;
    u32 hx = 0, h = 0, pf = 256, lp = 0, l0 = 0, l1 = 0;
#define HEAD_ASM "global_load_dwordx2 v[252:253], %[t], %[hdr] sc0 sc1\n\ts_mov_b64 exec, 1\n\tglobal_atomic_add v254, %[v16], %[one], %[tkt] sc0 sc1\n\ts_mov_b64 exec, -1\n\t"
    auto pull = [&]() {
#define PULL_ASM "global_load_dwordx3 v[244:246], %2, %1 offset:2048 sc0 sc1\n\tglobal_load_dwordx4 v[236:239], %0, %1 sc0 sc1\n\tglobal_load_dwordx4 v[240:243], %0, %1 offset:1024 sc0 sc1"
      if (__builtin_expect(kind == 3u, 1)) asm volatile(PULL_ASM :: "v"(o16), "s"(pk), "v"(o12) : "memory", LAND_REGS);
      else if (kind == 1u)
        asm volatile("global_load_dwordx4 v[236:239], %0, %1 sc0 sc1\n\tglobal_load_dwordx4 v[240:243], %0, %1 offset:1024 sc0 sc1\n\t"
                     "global_load_dwordx4 v[244:247], %0, %1 offset:2048 sc0 sc1\n\tglobal_load_dwordx4 v[248:251], %0, %1 offset:3072 sc0 sc1"
                     :: "v"(o16), "s"(pk) : "memory", LAND_REGS);
      else if (kind == 2u)
        asm volatile(PULL_ASM "\n\ts_lshr_b64 exec, -1, %4\n\tglobal_load_dword v247, %3, %1 offset:2816 sc0 sc1\n\ts_mov_b64 exec, -1"
                     :: "v"(o16), "s"(pk), "v"(o12), "v"(un * 4096u + lane * 4u), "s"(128u - ((hx >> 8) & 255u) - ((hx >> 16) & 255u)) : "memory", "scc", LAND_REGS);
    };
    const u32 scr = lbase + wave * PU_LDS;
    u32 vcall = call;
    asm volatile("" : "+v"(vcall));
    auto publish = [&]() {
      u32 t;
      asm volatile("v_lshl_or_b32 %0, %2, %6, %3\n\ts_lshl_b64 exec, 1, %1\n\tglobal_store_dword %0, %4, %5 sc0 sc1\n\ts_mov_b64 exec, -1"
                   : "=&v"(t) : "s"(up), "s"(up >> 6), "v"(vf), "v"(vcall), "s"(fb), "n"(V_FL + 2) : "memory", "scc"); };
    auto next = [&]() {
      u32 wl, hc, t;
      u = un; h = hx; od = o16;
      asm volatile("v_readfirstlane_b32 %[un], v254\n\ts_min_u32 %[un], %[un], " TK_MAX "\n\t" TK_ADD " %[un], %[un], %[nws]\n\ts_sub_u32 %[wl], %[un], %[wb]\n\t"
                   "v_readlane_b32 %[hx], v252, %[wl]\n\tv_readlane_b32 %[hc], v253, %[wl]\n\ts_add_u32 %[wb], %[un], 1\n\ts_max_u32 %[wb], %[wb], %[wmin]\n\t"
                   "v_lshl_add_u32 %[t], %[wb], 3, %[v8]\n\ts_or_b32 %[wl], %[un], %[tag]\n\t" HEAD_ASM
                   "v_lshl_add_u32 %[o16], %[un], 12, %[v16]\n\tv_lshl_add_u32 %[o12], %[un], 12, %[v12]\n\t"
                   "s_bfe_u32 %[kind], %[hx], 0x20018\n\ts_cmp_eq_u32 %[hc], %[wl]\n\ts_cselect_b32 %[kind], %[kind], 0"
                   : [un] "=&s"(un), [wl] "=&s"(wl), [hx] "=&s"(hx), [hc] "=&s"(hc), [wb] "+s"(wb), [kind] "=&s"(kind), [t] "=&v"(t), [o16] "=&v"(o16), [o12] "=&v"(o12)
                   : [nws] "s"(nws), [wmin] "s"(wmin), [tag] "s"(tag), [hdr] "s"(hdr), [tkt] "s"(tkt), [v8] "v"(v8), [v16] "v"(v16), [v12] "v"(v12), [one] "v"(1u)
                   : "memory", "scc", LAND_REGS); };
    for (;;) {
      WAIT_VM(0);
      publish();
      if (un >= 1024) break;
      for (u32 it = 1;; ++it) {
        const u64 v = __hip_atomic_load(lane == 1 ? (const u64*)tkt : (const u64*)(hdr + un * 8), __ATOMIC_RELAXED, __HIP_MEMORY_SCOPE_SYSTEM);
        hx = rfl((u32)v);
        wmin = (min(rdl((u32)v, 1), 1024u >> V_NS) << V_NS) + nws;
        if (rfl((u32)(v >> 32)) == (tag | un)) break;
        if ((it & 255u) == 0u && (rfl(__hip_atomic_load(tmo, __ATOMIC_RELAXED, __HIP_MEMORY_SCOPE_AGENT)) == call || late(t0))) {
          __hip_atomic_store(tmo, call, __ATOMIC_RELAXED, __HIP_MEMORY_SCOPE_AGENT);
          return;
        }
        __builtin_amdgcn_s_sleep(1);
      }
      kind = (hx >> 24) & 3u; o16 = un * 4096u + v16; o12 = un * 4096u + v12;
      pull();
      if (wb == 0u) {
        wb = wmin; fb = myws + WS_DUMMY + gi * 128; up = 0;
        asm volatile(HEAD_ASM :: [t] "v"(wb * 8u + v8), [hdr] "s"(hdr), [tkt] "s"(tkt), [v16] "v"(v16), [one] "v"(1u) : "memory", LAND_REGS);
      }
      WAIT_VM(0);
      for (;;) {
        next();
        if (V_FAST && kind == 3u && (h & 0x830000ffu) == (pf | 0x3000000u) && (hx & 255u) == pf) {
          u32 s0, s1, hc;
          asm volatile(FAST_ASM
            : [h0] "+s"(h), [h1] "+s"(hx), [u0] "+s"(u), [u1] "+s"(un), [o0] "+v"(od), [o1] "+v"(o16), [o12] "+v"(o12), [up] "+s"(up), [wb] "+s"(wb),
              [s0] "=&s"(s0), [s1] "=&s"(s1), [hc] "=&s"(hc)
            : [pk] "s"(pk), [xb] "s"(xb), [rdy] "s"(rdy), [hdr] "s"(hdr), [tkt] "s"(tkt), [nws] "s"(nws), [tag] "s"(tag), [pfk] "s"(pf | 0x3000000u),
              [lp] "s"(lp), [l1] "s"(l1), [l0] "v"(l0), [wa] "v"(scr + lane * 10u), [scr] "v"(scr), [vf] "v"(vf), [vcall] "v"(vcall), [v8] "v"(v8), [v16] "v"(v16), [v12] "v"(v12),
              [one] "v"(1u), [fsh] "n"(V_FL + 2), [k55] "s"(0x55555555u), [k124] "s"(0x124u), [kc31] "s"(0x0c0c0301u), [k7ff] "s"(0x7ffu), [k33] "s"(0x33333333u),
              [k80] "s"(0x80808080u), [kp0] "s"(0x05010400u), [kp1] "s"(0x07030602u)
            : "memory", "scc", FAST_CLOB, LAND_REGS);
          kind = hc == s0 ? (hx >> 24) & 3u : 0u;
        }
        u32x4 c[4];
        asm volatile("s_waitcnt vmcnt(7)\n\t"
                     "v_mov_b64 %0, v[236:237]\n\tv_mov_b64 %1, v[238:239]\n\tv_mov_b64 %2, v[240:241]\n\tv_mov_b64 %3, v[242:243]\n\t"
                     "v_mov_b64 %4, v[244:245]\n\tv_mov_b64 %5, v[246:247]\n\tv_mov_b64 %6, v[248:249]\n\tv_mov_b64 %7, v[250:251]"
                     : "=v"(c[0].lo), "=v"(c[0].hi), "=v"(c[1].lo), "=v"(c[1].hi), "=v"(c[2].lo), "=v"(c[2].hi), "=v"(c[3].lo), "=v"(c[3].hi)
                     :: "memory", LAND_REGS);
        pull();
        bool packed = true;
        if (__builtin_expect((h & 0x800000ffu) != pf, 0)) {
          packed = !(h >> 31);
          pf = h & 255u;
          const u32 f4 = pf * 0x01010101u;
          lp = exp_lut(f4, 0x00020100u); l0 = exp_lut(f4, 0xfbfaf903u); l1 = exp_lut(f4, 0xfffefdfcu);
        }
        if (__builtin_expect(packed, 1)) unpack_unit(c, h, lp, l0, l1, (LAS char*)smem + wave * PU_LDS, lane);
        asm volatile("" : "+v"(c[0]), "+v"(c[1]), "+v"(c[2]), "+v"(c[3]));
        auto finish = [&]() {
          publish();
          asm volatile("global_store_dwordx4 %0, %2, %1 sc0 sc1\n\tglobal_store_dwordx4 %0, %3, %1 offset:1024 sc0 sc1\n\t"
                       "global_store_dwordx4 %0, %4, %1 offset:2048 sc0 sc1\n\tglobal_store_dwordx4 %0, %5, %1 offset:3072 sc0 sc1\n\ts_nop 1"
                       :: "v"(od), "s"(xb), "v"(c[0]), "v"(c[1]), "v"(c[2]), "v"(c[3]) : "memory");
          up = u; fb = rdy; };
        if (__builtin_expect(kind == 0u, 0)) { WAIT_VM(0); finish(); asm volatile("; last"); break; }
        WAIT_VM(3);
        finish();
      }
    }
    return;
  }

  if (wave < 4) {
    const u32 w = wave;
    const u32x4 rw = mk_rsrc(P.w, (u32)KD * NL * 2);
    u32 pd, po = V_PF0 + (blockIdx.x * 4 + w) * V_PFL * 128;
    __builtin_amdgcn_s_setprio(2);
    const u32 r16 = lane & 15, g = lane >> 4, q = (lane >> 2) & 3, p = lane & 3;
    const u32 xa0 = lbase + A_OFF + r16 * 128 + 16 * (g ^ (r16 >> 1)), xa1 = xa0 ^ 64u;
    const u32 sw = (q >> 1) | ((g & 1) << 1);
    u32 wl[4];
#pragma unroll
    for (int nf = 0; nf < 4; ++nf) wl[nf] = lbase + W_OFF + w * 4096 + g * 1024 + q * 128 + ((nf ^ sw ^ xf) * 32) + 8 * p;
    u32 n = 21, bd = cbase + C_BAD;
    const u32 eb = lbase + W_OFF + 2 * WSLOT + w * 16384, ch = (((g & 1) ^ xf) * 2 + (g >> 1)) ^ (r16 & 7);
    u32 e0 = eb + r16 * 128 + ch * 16, e1 = e0 ^ 64u, er = eb + (lane >> 3) * 128 + ((lane ^ (lane >> 3)) & 7) * 16;
    u32 eo = ((rb * 128 + (lane >> 3)) * NL + cb * 256 + w * 64) * 2 + (lane & 7) * 16, el = eo + (rank * ML + rbi * 16 - rb * 128) * NL * 2;
    u32 ex = ((rank * ML + xrow + r16) * NL + cb * 256 + w * 64) * 2 + xf * 32 + g * 8;
    asm volatile("" : "+v"(e0), "+v"(e1), "+v"(er), "+v"(eo), "+v"(el), "+v"(ex));
#define CONS_IN [wl0] "v"(wl[0]), [wl1] "v"(wl[1]), [wl2] "v"(wl[2]), [wl3] "v"(wl[3]), [wh0] "v"(wl[0] + 65536u), [wh1] "v"(wl[1] + 65536u), \
      [wh2] "v"(wl[2] + 65536u), [wh3] "v"(wl[3] + 65536u), [pv] "v"(lane * V_PFL / 64 * 128), [rw] "s"(rw), [pi] "s"(896u * V_PFL * 128u), [xa0] "v"(xa0), [xa1] "v"(xa1)
    f32x32 a0, a1, a2, a3, xm; f32x16 a4, wm; f32x8 xl; f32x4 ax, xe;
#define CONS_OUT "=&{v[112:143]}"(a0), "=&{v[144:175]}"(a1), "=&{v[176:207]}"(a2), "=&{v[208:239]}"(a3), "=&{v[240:255]}"(a4), "=&{v[28:31]}"(ax), \
      "=&{v[60:75]}"(wm), "=&{v[76:107]}"(xm), "=&{v[36:43]}"(xl), "=&{v[32:35]}"(xe), [bd] "+v"(bd), [pd] "=&v"(pd), [po] "+s"(po), [n] "+s"(n)
    if (extra) asm volatile(CONS_ASM9X : CONS_OUT : CONS_IN : CONS_CLOB, "scc", "memory");
    else asm volatile(CONS_ASM9 : CONS_OUT : CONS_IN : CONS_CLOB, "scc", "memory");
    f32x4 a[36], wf[4], xq[9];
#pragma unroll
    for (int i = 0; i < 32; ++i) {
      a[i >> 2][i & 3] = a0[i]; a[8 + (i >> 2)][i & 3] = a1[i]; a[16 + (i >> 2)][i & 3] = a2[i]; a[24 + (i >> 2)][i & 3] = a3[i];
      if (i < 16) { a[32 + (i >> 2)][i & 3] = a4[i]; wf[i >> 2][i & 3] = wm[i]; }
      if (i < 28) xq[i >> 2][i & 3] = xm[i];
      if (i < 8) xq[7 + (i >> 2)][i & 3] = xl[i];
    }
#pragma unroll
    for (int i = 0; i < 4; ++i) wf[i][0] = bd == call ? __builtin_bit_cast(float, 0x7fc07fc0u) : wf[i][0];
    store_tile(a, wf, xq, (LAS char*)(size_t)e0, (LAS char*)(size_t)e1, (LAS char*)(size_t)er, A_OFF + ASLOT - W_OFF - 2 * WSLOT - (int)w * 14336, P.out, eo, el);
    if (extra) {
      ax = mma(wf[0], xe, ax);
      const u32x2 o = {pk_bf16(ax[0], ax[1]), pk_bf16(ax[2], ax[3])};
      asm volatile("global_store_dwordx2 %0, %1, %2 sc0 sc1\n\ts_nop 1" :: "v"(ex), "v"(o), "s"(P.out) : "memory");
    }
  } else if (wave != 6) {
    __builtin_amdgcn_s_setprio(3);
    const u32 e = wave == 7 ? 2 : wave - 4;
    const u32x4 rsrc = mk_rsrc(P.w, (u32)KD * NL * 2);
    const u32x4 rx = mk_rsrc(P.x, (u32)ML * KD * 2);
    const u32 r = lane >> 3;
    const u32 v0 = r * (NL * 2) + ((((lane & 7) >> 1) ^ ((r & 3) >> 1)) * 32) + (lane & 1) * 16, v1 = (v0 ^ 64u) + (8 * NL * 2 - 1024), v2 = v0 + 2 * (8 * NL * 2 - 1024), v3 = v1 + 2 * (8 * NL * 2 - 1024);
    const u32 o0 = r * (KD * 2) + 16 * ((lane & 7) ^ (lane >> 4)), o1 = (o0 ^ 64u) + 8 * KD * 2 - 1024;
    const u32 orow = rbi * 16, o2 = o0 + (xrow - orow) * (KD * 2) - 2048, o3 = o1 + (xrow - orow) * (KD * 2) - 2048;
    auto issue = [&](u32 t, u32 k) {
#pragma unroll
      for (int i = 0; i < 3; ++i) {
        const u32 gq = e * 3 + i;
        if (gq < 8) glds16x4<0>(lbase + W_OFF + k * 2 * WSLOT + gq * 4096, v0, v1, v2, v3, rsrc, (2 * t + (gq >> 2)) * 32 * (NL * 2) + cb * 512 + (gq & 3) * 128);
        else if (extra) glds16x4<0>(lbase + A_OFF + k * ASLOT + TILE, o0, o1, o2, o3, rx, orow * (KD * 2) + t * 128);
        else glds16x2(lbase + A_OFF + k * ASLOT + TILE, o0, o1, rx, orow * (KD * 2) + t * 128);
      }
    };
    issue(0, 0); issue(1, 1); issue(2, 2);
    const bool ten = e == 2 && !extra;
    if (ten) WAIT_VM(20); else WAIT_VM(24);
    __builtin_amdgcn_s_barrier();
    u32 k = 0;
    for (u32 t = 0; t < NT - 1; ++t) {
      if (t >= NT - 2) WAIT_VM(0); else if (ten) WAIT_VM(10); else WAIT_VM(12);
      __builtin_amdgcn_s_barrier();
      if (t + 3 < NT) issue(t + 3, k);
      if (++k == 3) k = 0;
    }
  } else {
    __builtin_amdgcn_s_setprio(3);
    const u32x4 rs = mk_rsrc(myws + WS_X, (u32)(WS_NEED - WS_X));
    const char* fl = myws + WS_RDY + (size_t)rb * NT * FS;
    const u32 vl = lane * 16;
    u32 need = 0, issued = 0, want = call, ld = lbase + A_OFF, so = (u32)XPAR + ((par * 8 + (rb >> 2)) * U0 + (rb & 3) * 4) * 4096;
    const u32 jump = ((rb >> 2) * 1024 + (rb & 3) * 4) * 4096 - so + 65536;
    u32x4 f = {0u, 0u, 0u, 0u};
    u64 t0 = 0;
#define A_LD4(v) "buffer_load_dwordx4 %[" #v "], %[rs], %[so] offen sc0 sc1 lds\n\tbuffer_load_dwordx4 %[" #v "], %[rs], %[so] offen offset:1024 sc0 sc1 lds\n\t" \
                 "buffer_load_dwordx4 %[" #v "], %[rs], %[so] offen offset:2048 sc0 sc1 lds\n\tbuffer_load_dwordx4 %[" #v "], %[rs], %[so] offen offset:3072 sc0 sc1 lds\n\t"
#define A_TILE "s_mov_b32 m0, %[ld]\n\ts_add_u32 %[iss], %[iss], 1\n\t" A_LD4(v0) \
               "s_add_u32 m0, m0, 0x1000\n\ts_add_u32 %[ld], %[ld], %[aslot]\n\t" A_LD4(v1) \
               "s_add_u32 m0, m0, 0x1000\n\ts_cmp_eq_u32 %[ld], %[lend]\n\ts_cselect_b32 %[ld], %[lbeg], %[ld]\n\t" A_LD4(v2) \
               "s_add_u32 m0, m0, 0x1000\n\ts_cmp_eq_u32 %[iss], %[tp]\n\ts_cselect_b32 %[t], %[jump], 0x10000\n\t" A_LD4(v3) \
               "s_add_u32 %[so], %[so], %[t]\n\t"
#define A_POLL "s_lshl_b64 exec, " V_WIN ", %[iss]\n\tglobal_load_dwordx4 %[f], %[vp], %[fl] sc0 sc1\n\ts_mov_b64 exec, -1\n\t"
#define A_REFRESH "v_min3_u32 %[m], v20, v21, v22\n\tv_min_u32 %[m], %[m], v23\n\tv_cmp_gt_u32 vcc, %[want], %[m]\n\ts_ff1_i32_b64 %[av], vcc\n\ts_min_u32 %[av], %[av], 64\n\ts_cmp_gt_u32 %[av], %[iss]\n\t"
    for (;;) {
      u32 av, t, m, spin = 256;
      asm volatile(
        "La_top_%=:\n\t" A_POLL
        "La_ref_%=:\n\t" A_REFRESH
        "s_cbranch_scc0 La_none_%=\n\t" A_TILE
        "s_add_u32 %[need], %[need], 1\n\ts_cmp_gt_u32 %[iss], %[need]\n\ts_cbranch_scc0 La_late_%=\n\t"
        "s_waitcnt vmcnt(17)\n\ts_barrier\n\ts_branch La_top_%=\n"
        "La_late_%=:\n\t"
        "s_cmp_gt_u32 %[av], %[iss]\n\ts_cbranch_scc0 La_one_%=\n\t" A_TILE
        "s_waitcnt vmcnt(16)\n\ts_barrier\n\ts_branch La_top_%=\n"
        "La_one_%=:\n\ts_cmp_eq_u32 %[need], 64\n\ts_cbranch_scc1 La_out_%=\n\ts_waitcnt vmcnt(0)\n\ts_barrier\n\ts_branch La_top_%=\n"
        "La_none_%=:\n\ts_cmp_gt_u32 %[iss], %[need]\n\ts_cbranch_scc0 La_hard_%=\n\t"
        "s_add_u32 %[need], %[need], 1\n\ts_cmp_eq_u32 %[need], 64\n\ts_cbranch_scc1 La_out_%=\n\t"
        "s_waitcnt vmcnt(0)\n\t" A_REFRESH "s_cbranch_scc0 La_skip_%=\n\t" A_TILE
        "La_skip_%=:\n\t"
        "s_barrier\n\ts_branch La_top_%=\n"
        "La_hard_%=:\n\t"
        "s_waitcnt vmcnt(0)\n\t"
        A_REFRESH "s_cbranch_scc1 La_ref_%=\n\t"
        "s_sub_u32 %[spin], %[spin], 1\n\ts_cbranch_scc0 La_top_%=\n"
        "La_out_%=:\n\ts_mov_b64 exec, 1\n\tglobal_load_dword %[m], %[v0], %[tmo] sc1\n\ts_mov_b64 exec, -1\n\ts_waitcnt vmcnt(0)"
        : [need] "+s"(need), [iss] "+s"(issued), [ld] "+s"(ld), [so] "+s"(so), [f] "+{v[20:23]}"(f), [av] "=&s"(av), [t] "=&s"(t), [m] "=&v"(m), [spin] "+s"(spin)
        : [rs] "s"(rs), [fl] "s"(fl), [vp] "v"(lane * FS), [v0] "v"(vl), [v1] "v"(vl + 4096u), [v2] "v"(vl + 8192u), [v3] "v"(vl + 12288u), [want] "s"(want), [jump] "s"(jump), [tmo] "s"(tmo),
          [lbeg] "s"(lbase + A_OFF), [lend] "s"(lbase + A_OFF + 3 * ASLOT), [aslot] "n"(ASLOT), [tp] "n"(TP)
        : "memory", "scc", "vcc");
      const bool dead = rfl(m) == call;
      if (need == NT) { if (dead) lds_wr1(cbase + C_BAD, call); break; }
      if (dead || late(t0)) { __hip_atomic_store(tmo, call, __ATOMIC_RELAXED, __HIP_MEMORY_SCOPE_AGENT); want = 0; WAIT_VM(0); }
    }
    asm volatile("s_waitcnt vmcnt(0) lgkmcnt(0)" ::: "memory");
    __builtin_amdgcn_s_barrier();
  }
}

extern "C" void kernel_launch(int rank, int nranks, ncclComm_t comm, hipStream_t stream,
                              void* const* d_in, const int* in_sizes, int n_in, void* d_out, int out_size,
                              const int* in_dtypes, int out_dtype, void* const* d_out_all,
                              void* const* d_ws_all, size_t ws_bytes) {
  if (nranks != NR || ws_bytes < WS_NEED || (unsigned)rank >= (unsigned)NR) { fprintf(stderr, "unexpected configuration\n"); return; }
  hipError_t e = hipFuncSetAttribute((const void*)ag_gemm_kernel, hipFuncAttributeMaxDynamicSharedMemorySize, LDS_BYTES);
  if (e != hipSuccess) { fprintf(stderr, "attr: %s\n", hipGetErrorString(e)); return; }
  Params P;
  P.x = (const char*)d_in[0]; P.w = (const char*)d_in[1]; P.out = (char*)d_out; P.rank = rank; P.pad = 0;
  P.my = (char*)d_ws_all[rank];
  for (int p = 0; p < NR - 1; ++p) P.peer[p] = (char*)d_ws_all[(rank + 1 + p) & 7];
  hipLaunchKernelGGL(ag_gemm_kernel, dim3(256), dim3(512), LDS_BYTES, stream, P);
}
```
